```python
import numpy as np

kernel_name = "gpu_a2a_gemm_m4096_k4096_n4096_e4m3e4m3_dequant"


ABSMAX_RTOL = 0.03

COMPUTE_DTYPE = "float32"
EPILOGUE = "dequant"
IN_QUANT = None
OUT_QUANT = None
IN_FP8 = "e4m3e4m3"
IN_DTYPES = {"x": "e4m3", "w_mat": "e4m3",
             "scale_x": "float32", "scale_w": "float32"}
WORLD_SIZE = 8

def _snap_fp8(x, mant_bits, max_abs, min_normal):
    a = np.abs(x.astype(np.float64))
    _m, _e = np.frexp(a)
    q = np.where(a >= min_normal, np.ldexp(1.0, _e - 1 - mant_bits),
                 min_normal * 2.0 ** -mant_bits)
    snapped = np.minimum(np.round(a / q) * q, max_abs)
    return (np.sign(x) * snapped).astype(np.float32)

def setup_inputs():

    x = _snap_fp8(np.random.randn(4096, 4096).astype(np.float32), 3, 448.0, 2.0 ** -6)
    w_mat = _snap_fp8(np.random.randn(4096, 4096).astype(np.float32), 3, 448.0, 2.0 ** -6)
    scale_x = (0.02 * (1.0 + np.random.rand(1))).astype(np.float32)
    scale_w = (0.02 * (1.0 + np.random.rand(1))).astype(np.float32)
    return {"x": x, "w_mat": w_mat, "scale_x": scale_x, "scale_w": scale_w}

def reference(x, w_mat, scale_x, scale_w):
    acc = (x @ w_mat).astype(np.float32)
    y = (acc * (scale_x * scale_w)).astype(np.float32)
    return y
```

```cpp
#include <hip/hip_runtime.h>
#include <rccl/rccl.h>
#include <cstdio>
#include <cstdint>

#define HIP_CHECK(expr) do { hipError_t _e = (expr); if (_e != hipSuccess) { \
    fprintf(stderr, "HIP error %s at %s:%d\n", hipGetErrorString(_e), __FILE__, __LINE__); \
    return; } } while (0)

typedef int v2i __attribute__((ext_vector_type(2)));
typedef int v4i __attribute__((ext_vector_type(4)));
typedef unsigned v4u __attribute__((ext_vector_type(4)));
typedef int v8i __attribute__((ext_vector_type(8)));
typedef float v4f __attribute__((ext_vector_type(4)));
#define LAS __attribute__((address_space(3)))
#define GAS __attribute__((address_space(1)))

namespace {

constexpr int NR = 8;
constexpr int MS = 512;
constexpr int KC = 512;
constexpr int NN = 4096;
constexpr int KK = 4096;
constexpr int NWG = 256;
constexpr int NTHREADS = 512;
constexpr int NSTEP = 32;
constexpr int NREM = 28;

#define NSLOT 18
constexpr int SLOT_BYTES = 8192;
constexpr int CTRL_OFF = NSLOT * SLOT_BYTES;
constexpr int L_READY = CTRL_OFF;
constexpr int L_PROG = CTRL_OFF + 16;
constexpr int L_FULL = CTRL_OFF + 32;
constexpr int L_GO = CTRL_OFF + 48;
constexpr int LDS_BYTES = CTRL_OFF + 64;
static_assert(LDS_BYTES <= 160 * 1024, "LDS");

constexpr size_t WS_FLAG = 0;
constexpr size_t WS_TMO = 8192;
constexpr size_t WS_CNT = 16384;
constexpr size_t WS_DBG = 65536;
constexpr size_t WS_A = 1u << 20;
constexpr size_t A_BYTES = (size_t)MS * KK;
constexpr size_t WS_NEED = WS_A + 2 * A_BYTES;

constexpr unsigned long long DEADLINE_TICKS = 50000000ull;
constexpr int READY_BAD = 0x100, READY_PAR = 0x200;

struct Params {
    const unsigned char* x;
    const unsigned char* w;
    const float* sx;
    const float* sw;
    float* out;
    unsigned char* ws[NR];
    int rank;
    int pad;
};

#define STAMP(slot) do {} while (0)

__device__ __forceinline__ unsigned lds_ld(LAS unsigned char* L, int off) {
    return *(volatile LAS unsigned*)(L + off);
}
__device__ __forceinline__ void lds_st(LAS unsigned char* L, int off, unsigned v) {
    *(volatile LAS unsigned*)(L + off) = v;
}
#define CBAR() asm volatile("" ::: "memory")

__device__ __forceinline__ v8i cat8(v4u a, v4u b) {
    v8i r;
    r[0] = a[0]; r[1] = a[1]; r[2] = a[2]; r[3] = a[3];
    r[4] = b[0]; r[5] = b[1]; r[6] = b[2]; r[7] = b[3];
    return r;
}

}

__global__ void __launch_bounds__(NTHREADS, 2) a2a_gemm_kernel(Params p) {
    extern __shared__ __attribute__((aligned(16))) unsigned char smem[];
    LAS unsigned char* L = (LAS unsigned char*)smem;
    const int tid = threadIdx.x;
    const int wave = __builtin_amdgcn_readfirstlane(tid >> 6);
    const int lane = tid & 63;
    const int b = blockIdx.x;
    const int rank = p.rank;
    unsigned char* myws = p.ws[rank];
    const int mt = (b >> 3) & 3;
    const int nt = (b & 7) * 8 + (b >> 5);
    const int n0 = nt * 64;

    if (wave == 0) {
        __builtin_amdgcn_s_setprio(3);
        STAMP(0);
        const unsigned dst = (unsigned)b >> 5;
        unsigned* cntp = (unsigned*)(myws + WS_CNT + (size_t)b * 128);
        const unsigned char* src = p.x + (size_t)(b * 16 + (lane & 15)) * KC + (lane >> 4) * 16;
        v4u d[8];
        unsigned cnt;
        asm volatile(
            "global_load_dword %8, %10, off sc1\n\t"
            "global_load_dwordx4 %0, %9, off\n\t"
            "global_load_dwordx4 %1, %9, off offset:64\n\t"
            "global_load_dwordx4 %2, %9, off offset:128\n\t"
            "global_load_dwordx4 %3, %9, off offset:192\n\t"
            "global_load_dwordx4 %4, %9, off offset:256\n\t"
            "global_load_dwordx4 %5, %9, off offset:320\n\t"
            "global_load_dwordx4 %6, %9, off offset:384\n\t"
            "global_load_dwordx4 %7, %9, off offset:448\n\t"
            "s_barrier\n\t"
            "s_waitcnt vmcnt(0)"
            : "=&v"(d[0]), "=&v"(d[1]), "=&v"(d[2]), "=&v"(d[3]), "=&v"(d[4]), "=&v"(d[5]),
              "=&v"(d[6]), "=&v"(d[7]), "=&v"(cnt)
            : "v"(src), "v"(cntp) : "memory");
        const unsigned call = cnt + 1;
        if (dst != (unsigned)rank && dst < (unsigned)NR) {
            const int m = (b >> 3) & 3, s = b & 7;
            unsigned char* pw = p.ws[dst];
            unsigned char* dp = pw + WS_A + (size_t)(call & 1) * A_BYTES +
                                (size_t)((rank * 4 + m) * 4) * 16384 + s * 2048 + lane * 16;
#pragma unroll
            for (int g = 0; g < 4; ++g) {
                unsigned char* q = dp + g * 16384;
                asm volatile("global_store_dwordx4 %0, %1, off sc0 sc1\n\t"
                             "global_store_dwordx4 %0, %2, off offset:1024 sc0 sc1\n\ts_nop 1"
                             :: "v"(q), "v"(d[2 * g]), "v"(d[2 * g + 1]) : "memory");
            }
            lds_st(L, L_GO, 1u);
            STAMP(1);
            unsigned* flag = (unsigned*)(pw + WS_FLAG + (size_t)(m * 8 + rank) * 128) + s;
            const unsigned v0 = cnt * 4;
            asm volatile("s_waitcnt vmcnt(6)\n\t"
                         "global_store_dword %0, %1, off sc0 sc1\n\t"
                         "s_waitcnt vmcnt(5)\n\t"
                         "global_store_dword %0, %2, off sc0 sc1\n\t"
                         "s_waitcnt vmcnt(4)\n\t"
                         "global_store_dword %0, %3, off sc0 sc1\n\t"
                         "s_waitcnt vmcnt(3)\n\t"
                         "global_store_dword %0, %4, off sc0 sc1"
                         :: "v"(flag), "v"(v0 + 1), "v"(v0 + 2), "v"(v0 + 3), "v"(v0 + 4) : "memory");
        } else {
            lds_st(L, L_GO, 1u);
        }
        if (lane == 0) __hip_atomic_store(cntp, call, __ATOMIC_RELAXED, __HIP_MEMORY_SCOPE_AGENT);
        return;
    }

    if (wave == 1) {
        unsigned* cntp = (unsigned*)(myws + WS_CNT + (size_t)b * 128) + 1;
        const unsigned cnt = __hip_atomic_load(cntp, __ATOMIC_RELAXED, __HIP_MEMORY_SCOPE_AGENT);
        if (lane < 16) *(volatile LAS unsigned*)(L + CTRL_OFF + lane * 4) = 0u;
        asm volatile("s_waitcnt lgkmcnt(0)\n\ts_barrier" ::: "memory");
        __builtin_amdgcn_s_setprio(2);
        const unsigned call = cnt + 1;
        if (lane == 0) __hip_atomic_store(cntp, call, __ATOMIC_RELAXED, __HIP_MEMORY_SCOPE_AGENT);
        const int s = lane >> 3, q = lane & 7;
        const int src = (rank + 1 + q) & 7;
        unsigned* tmo = (unsigned*)(myws + WS_TMO);
        unsigned* fp = q < 7 ? (unsigned*)(myws + WS_FLAG + (size_t)(mt * 8 + src) * 128) + s : tmo;
        const unsigned base = cnt * 4;
        const unsigned par = (call & 1) ? READY_PAR : 0;
        const unsigned long long t0 = __builtin_amdgcn_s_memrealtime();
        int last = -1;
        for (;;) {
            const unsigned v = __hip_atomic_load(fp, __ATOMIC_RELAXED, __HIP_MEMORY_SCOPE_SYSTEM);
            int d = (int)(v - base);
            d = d < 0 ? 0 : (d > 4 ? 4 : d);
            int r = q < 7 ? d * 7 + q : NREM;
            r = r > NREM ? NREM : r;
            r = min(r, __builtin_amdgcn_update_dpp(r, r, 0x128, 0xf, 0xf, false));
            r = min(r, __builtin_amdgcn_update_dpp(r, r, 0x124, 0xf, 0xf, false));
            r = min(r, __builtin_amdgcn_update_dpp(r, r, 0x122, 0xf, 0xf, false));
            r = min(r, __builtin_amdgcn_update_dpp(r, r, 0x121, 0xf, 0xf, false));
            const bool done = __all(r >= NREM);
            const bool dead = __any(q == 7 && v == call) ||
                              (__builtin_amdgcn_s_memrealtime() - t0 > DEADLINE_TICKS);
            if (dead) {
                if (lane == 0) __hip_atomic_store(tmo, call, __ATOMIC_RELAXED, __HIP_MEMORY_SCOPE_AGENT);
                r = NREM | READY_BAD;
            }
            if (r != last && (lane & 15) == 0)
                __hip_atomic_store((LAS unsigned*)(L + L_READY) + (lane >> 4), (unsigned)r | par,
                                   __ATOMIC_RELEASE, __HIP_MEMORY_SCOPE_WORKGROUP);
            last = r;
            if (done || dead) break;
            __builtin_amdgcn_s_sleep(1);
        }
        return;
    }

    if (wave < 4) {
        const int ld = wave - 2;
        __builtin_amdgcn_s_barrier();
        __builtin_amdgcn_s_setprio(1);
        const __amdgpu_buffer_rsrc_t rw =
            __builtin_amdgcn_make_buffer_rsrc((void*)p.w, (short)0, KK * NN, 0x00020000);
        const int voff = (lane >> 2) * NN + (((lane & 3) ^ (((lane >> 4) & 1) | (ld << 1))) * 16);
        while (lds_ld(L, L_GO) == 0u) __builtin_amdgcn_s_sleep(1);
        CBAR();
        constexpr int LAG = 8;
        int slot = 0;
        for (int i = 0; i < NSTEP; ++i) {
            if (i >= NSLOT) {
                const unsigned need = (unsigned)(i - NSLOT + 1);
                for (;;) {
                    const v4u pr = *(volatile LAS v4u*)(L + L_PROG);
                    const unsigned mn = min(min(pr[0], pr[1]), min(pr[2], pr[3]));
                    if (__builtin_amdgcn_readfirstlane(mn) >= need) break;
                    __builtin_amdgcn_s_sleep(1);
                }
                CBAR();
            }
            int k0;
            if (i < 4) k0 = rank * KC + i * 128;
            else { const int j = i - 4, g = j / 7, q = j - g * 7; k0 = ((rank + 1 + q) & 7) * KC + g * 128; }
            const int soff = (k0 + ld * 16) * NN + n0;
            LAS unsigned char* dstl = L + slot * SLOT_BYTES + ld * 1024;
#pragma unroll
            for (int u = 0; u < 4; ++u)
                __builtin_amdgcn_raw_ptr_buffer_load_lds(rw, (LAS void*)(dstl + u * 2048), 16, voff,
                                                         soff + u * 32 * NN, 0, 0);
            slot = slot + 1 == NSLOT ? 0 : slot + 1;
            if (i >= LAG) {
                asm volatile("s_waitcnt vmcnt(32)" ::: "memory");
                lds_st(L, L_FULL + ld * 4, (unsigned)(i - LAG + 1));
            }
        }
#define DRAIN(n) asm volatile("s_waitcnt vmcnt(" #n ")" ::: "memory"); \
        lds_st(L, L_FULL + ld * 4, (unsigned)(NSTEP - (n) / 4));
        DRAIN(28) DRAIN(24) DRAIN(20) DRAIN(16) DRAIN(12) DRAIN(8) DRAIN(4) DRAIN(0)
#undef DRAIN
        if (ld == 0) STAMP(16);
        return;
    }

    const int cw = wave - 4;
    const __amdgpu_buffer_rsrc_t rx =
        __builtin_amdgcn_make_buffer_rsrc((void*)p.x, (short)0, NR * MS * KC, 0x00020000);
    const __amdgpu_buffer_rsrc_t ra =
        __builtin_amdgcn_make_buffer_rsrc((void*)(myws + WS_A), (short)0, (int)(2 * A_BYTES), 0x00020000);
    v8i a[4][2];
    {
        const int voff = (lane & 15) * KC + (lane >> 4) * 16;
        const int soff = (rank * MS + mt * 128 + cw * 32) * KC;
#pragma unroll
        for (int g = 0; g < 4; ++g)
#pragma unroll
            for (int s = 0; s < 2; ++s)
                a[g][s] = cat8(
                    __builtin_amdgcn_raw_buffer_load_b128(rx, voff + g * 128, soff + s * 16 * KC, 0),
                    __builtin_amdgcn_raw_buffer_load_b128(rx, voff + g * 128 + 64, soff + s * 16 * KC, 0));
    }
    const float scale = p.sx[0] * p.sw[0];
    __builtin_amdgcn_s_barrier();

    v4f acc[2][4];
#pragma unroll
    for (int s = 0; s < 2; ++s)
#pragma unroll
        for (int t = 0; t < 4; ++t) acc[s][t] = v4f{0.f, 0.f, 0.f, 0.f};

    const int g4 = lane >> 4, q8 = (lane & 15) >> 1;
    const int trb = (g4 * 16 + q8) * 64 + ((((q8 >> 2) & 1) | ((g4 & 1) << 1)) * 16) + (lane & 1) * 8;
    int full = 0, ready = 0, slot = 0;
    v8i bf[2][4];

    auto wait_full = [&](int i) {
        if (full <= i) {
            for (;;) {
                const v2i f = *(volatile LAS v2i*)(L + L_FULL);
                full = __builtin_amdgcn_readfirstlane(min(f[0], f[1]));
                if (full > i) break;
                __builtin_amdgcn_s_sleep(1);
            }
        }
        CBAR();
    };
    auto read_b = [&](v8i (&dst)[4]) {
        LAS unsigned char* sb = L + slot * SLOT_BYTES;
#pragma unroll
        for (int t = 0; t < 4; ++t) {
            LAS unsigned char* q = sb + (trb ^ (t * 16));
            const v2i r0 = __builtin_amdgcn_ds_read_tr8_b64_v2i32((LAS v2i*)(q));
            const v2i r1 = __builtin_amdgcn_ds_read_tr8_b64_v2i32((LAS v2i*)(q + 512));
            const v2i r2 = __builtin_amdgcn_ds_read_tr8_b64_v2i32((LAS v2i*)(q + 4096));
            const v2i r3 = __builtin_amdgcn_ds_read_tr8_b64_v2i32((LAS v2i*)(q + 4608));
            v8i r;
            r[0] = r0[0]; r[1] = r0[1]; r[2] = r1[0]; r[3] = r1[1];
            r[4] = r2[0]; r[5] = r2[1]; r[6] = r3[0]; r[7] = r3[1];
            dst[t] = r;
        }
        slot = slot + 1 == NSLOT ? 0 : slot + 1;
    };
    auto mma = [&](const v8i (&af)[2], const v8i (&bb)[4]) {
#pragma unroll
        for (int t = 0; t < 4; ++t)
#pragma unroll
            for (int s = 0; s < 2; ++s)
                acc[s][t] = __builtin_amdgcn_mfma_scale_f32_16x16x128_f8f6f4(bb[t], af[s], acc[s][t],
                                                                           0, 0, 0, 0, 0, 0);
    };
    auto release = [&](int n) {
        asm volatile("s_waitcnt lgkmcnt(0)" ::: "memory");
        lds_st(L, L_PROG + cw * 4, (unsigned)n);
    };
    auto load_a = [&](int j, v8i (&dst)[2]) {
        if ((ready & 0xff) <= j) {
            for (;;) {
                ready = __builtin_amdgcn_readfirstlane((int)__hip_atomic_load(
                    (LAS unsigned*)(L + L_READY) + cw, __ATOMIC_ACQUIRE, __HIP_MEMORY_SCOPE_WORKGROUP));
                if ((ready & 0xff) > j) break;
                __builtin_amdgcn_s_sleep(1);
            }
        }
        CBAR();
        const int g = j / 7, q = j - g * 7, src = (rank + 1 + q) & 7;
        const int soff = ((ready & READY_PAR) ? (int)A_BYTES : 0) + ((src * 4 + mt) * 4 + g) * 16384 + cw * 4096;
        const int voff = lane * 16;
        dst[0] = cat8(__builtin_amdgcn_raw_buffer_load_b128(ra, voff, soff, 17),
                      __builtin_amdgcn_raw_buffer_load_b128(ra, voff + 1024, soff, 17));
        dst[1] = cat8(__builtin_amdgcn_raw_buffer_load_b128(ra, voff + 2048, soff, 17),
                      __builtin_amdgcn_raw_buffer_load_b128(ra, voff + 3072, soff, 17));
        CBAR();
    };

    if (cw == 0) STAMP(24);
    wait_full(0); read_b(bf[0]);
#pragma unroll
    for (int i = 0; i < 4; ++i) {
        release(i + 1);
        wait_full(i + 1); read_b(bf[(i + 1) & 1]);
        mma(a[i], bf[i & 1]);
    }
    if (cw == 0) STAMP(25);
    load_a(0, a[0]); load_a(1, a[1]); load_a(2, a[2]);
    for (int j0 = 0; j0 < NREM; j0 += 4) {
#pragma unroll
        for (int u = 0; u < 4; ++u) {
            const int j = j0 + u;
            if (j + 3 < NREM) load_a(j + 3, a[(u + 3) & 3]);
            release(j + 5);
            if (j + 5 < NSTEP) { wait_full(j + 5); read_b(bf[(u + 1) & 1]); }
            mma(a[u], bf[u & 1]);
        }
    }

    const float sc = (ready & READY_BAD) ? __builtin_nanf("") : scale;
    float* o = p.out + (size_t)(mt * 128 + cw * 32 + (lane & 15)) * NN + n0 + g4 * 4;
#pragma unroll
    for (int s = 0; s < 2; ++s)
#pragma unroll
        for (int t = 0; t < 4; ++t) {
            v4f v = acc[s][t] * sc;
            *(v4f*)(o + (size_t)s * 16 * NN + t * 16) = v;
        }
}

extern "C" void kernel_launch(int rank, int nranks, ncclComm_t comm, hipStream_t stream,
                              void* const* d_in, const int* in_sizes, int n_in,
                              void* d_out, int out_size, const int* in_dtypes, int out_dtype,
                              void* const* d_out_all, void* const* d_ws_all, size_t ws_bytes) {
    (void)comm; (void)in_dtypes; (void)out_dtype; (void)d_out_all;
    if (nranks != NR || (unsigned)rank >= (unsigned)NR || n_in < 4 || ws_bytes < WS_NEED ||
        out_size != MS * NN || in_sizes[0] != NR * MS * KC || in_sizes[1] != KK * NN) {
        fprintf(stderr, "a2a_gemm: unexpected problem shape\n");
        return;
    }
    HIP_CHECK(hipFuncSetAttribute((const void*)a2a_gemm_kernel,
                                  hipFuncAttributeMaxDynamicSharedMemorySize, LDS_BYTES));
    Params p;
    p.x = (const unsigned char*)d_in[0];
    p.w = (const unsigned char*)d_in[1];
    p.sx = (const float*)d_in[2];
    p.sw = (const float*)d_in[3];
    p.out = (float*)d_out;
    for (int i = 0; i < NR; ++i) p.ws[i] = (unsigned char*)d_ws_all[i];
    p.rank = rank;
    p.pad = 0;
    hipLaunchKernelGGL(a2a_gemm_kernel, dim3(NWG), dim3(NTHREADS), LDS_BYTES, stream, p);
    HIP_CHECK(hipGetLastError());
}
```

```cpp
#include <hip/hip_runtime.h>
#include <rccl/rccl.h>
#include <cstdio>
#include <cstdint>

#define HIP_CHECK(expr) do { hipError_t _e = (expr); if (_e != hipSuccess) { \
    fprintf(stderr, "HIP error %s at %s:%d\n", hipGetErrorString(_e), __FILE__, __LINE__); \
    return; } } while (0)

#define LAS __attribute__((address_space(3)))

#define T0 17
#define XD 3
#define NW 7
#define NA 5
#define NBAR 44
#define KNOWN 0x7fffff0000000fffull
#define W_SCHEDULE \
    W_HALF(0, 0, 0); W_HALF(0, 1, 0); W_HALF(1, 0, 1); W_HALF(1, 1, 1); W_HALF(2, 0, 2); W_HALF(2, 1, 2); \
    WVX(16); BAR(); W_HALF(3, 0, 3); W_HALF(3, 1, 3); WVX(16); BAR(); \
    W_HALF(4, 0, 4); W_HALF(4, 1, 4); WVX(16); BAR(); W_HALF(5, 0, 5); W_HALF(5, 1, 5); \
    WVX(16); BAR(); W_HALF(6, 0, 6); W_HALF(6, 1, 6); WVX(16); BAR(); \
    W_HALF(7, 0, 0); W_HALF(7, 1, 0); WVX(16); BAR(); W_HALF(8, 0, 1); W_HALF(8, 1, 1); \
    WVX(16); BAR(); W_HALF(9, 0, 2); W_HALF(9, 1, 2); WVX(16); BAR(); \
    W_HALF(10, 0, 3); W_HALF(10, 1, 3); WVX(16); BAR(); W_HALF(11, 0, 4); W_HALF(11, 1, 4); \
    WVX(16); BAR(); W_HALF(12, 0, 5); W_HALF(12, 1, 5); WVX(16); BAR(); \
    W_HALF(13, 0, 6); W_HALF(13, 1, 6); WVX(16); BAR(); W_HALF(14, 0, 0); W_HALF(14, 1, 0); \
    WVX(16); BAR(); W_HALF(15, 0, 1); W_HALF(15, 1, 1); WVX(16); BAR(); \
    W_HALF(16, 0, 2); W_HALF(16, 1, 2); WVX(16); BAR(); W_HALF(17, 0, 3); W_HALF(17, 1, 3); \
    BAR(); W_HALF(18, 0, 4); W_HALF(18, 1, 4); BAR(); W_HALF(19, 0, 5); W_HALF(19, 1, 5); \
    BAR(); W_HALF(20, 0, 6); W_HALF(20, 1, 6); WVX(40); BAR(); W_HALF(21, 0, 0); \
    WVX(36); BAR(); W_HALF(21, 1, 0); WVX(32); BAR(); W_HALF(22, 0, 1); \
    WVX(28); BAR(); W_HALF(22, 1, 1); WVX(24); BAR(); W_HALF(23, 0, 2); \
    WVX(20); BAR(); W_HALF(23, 1, 2); WVX(16); BAR(); W_HALF(24, 0, 3); \
    WVX(12); BAR(); W_HALF(24, 1, 3); WVX(8); BAR(); W_HALF(25, 0, 4); \
    BAR(); W_HALF(25, 1, 4); WVX(8); BAR(); W_HALF(26, 0, 5); BAR(); \
    W_HALF(26, 1, 5); WVX(8); BAR(); W_HALF(27, 0, 6); BAR(); W_HALF(27, 1, 6); \
    WVX(8); BAR(); W_HALF(28, 0, 0); BAR(); W_HALF(28, 1, 0); WVX(8); \
    BAR(); W_HALF(29, 0, 1); BAR(); W_HALF(29, 1, 1); WVX(8); BAR(); \
    W_HALF(30, 0, 2); BAR(); W_HALF(30, 1, 2); WVX(8); BAR(); W_HALF(31, 0, 3); \
    BAR(); W_HALF(31, 1, 3); WVX(8); BAR(); BAR(); WVX(0);
#define COMPUTE_ASM \
    "s_waitcnt lgkmcnt(0)\n\t" \
    "s_load_dword %[sx], %[psx], 0x0\n\t" \
    "s_load_dword %[sw], %[psw], 0x0\n\t" \
    "s_barrier\n\t" \
    "s_waitcnt lgkmcnt(0)\n\t" \
    "s_barrier\n\t" \
    "ds_read_b128 v[48:51], %[ab0] offset:0\n\t" \
    "ds_read_b64_tr_b8 v[80:81], %[wb0] offset:0\n\t" \
    "ds_read_b64_tr_b8 v[82:83], %[wb0] offset:1024\n\t" \
    "ds_read_b128 v[52:55], %[ab1] offset:0\n\t" \
    "ds_read_b64_tr_b8 v[84:85], %[wb0] offset:8192\n\t" \
    "ds_read_b64_tr_b8 v[86:87], %[wb0] offset:9216\n\t" \
    "ds_read_b128 v[56:59], %[ab0] offset:2048\n\t" \
    "ds_read_b64_tr_b8 v[88:89], %[wb1] offset:0\n\t" \
    "ds_read_b64_tr_b8 v[90:91], %[wb1] offset:1024\n\t" \
    "ds_read_b128 v[60:63], %[ab1] offset:2048\n\t" \
    "ds_read_b64_tr_b8 v[92:93], %[wb1] offset:8192\n\t" \
    "ds_read_b64_tr_b8 v[94:95], %[wb1] offset:9216\n\t" \
    "ds_read_b128 v[64:67], %[ab0] offset:4096\n\t" \
    "ds_read_b128 v[68:71], %[ab1] offset:4096\n\t" \
    "ds_read_b128 v[72:75], %[ab0] offset:6144\n\t" \
    "ds_read_b128 v[76:79], %[ab1] offset:6144\n\t" \
    "s_waitcnt lgkmcnt(10)\n\t" \
    "v_mfma_f32_16x16x128_f8f6f4 v[16:19], v[80:87], v[48:55], 0\n\t" \
    "s_barrier\n\t" \
    "s_waitcnt lgkmcnt(4)\n\t" \
    "v_mfma_f32_16x16x128_f8f6f4 v[20:23], v[88:95], v[48:55], 0\n\t" \
    "ds_read_b128 v[48:51], %[ab0] offset:8192\n\t" \
    "ds_read_b64_tr_b8 v[96:97], %[wb0] offset:16384\n\t" \
    "ds_read_b64_tr_b8 v[98:99], %[wb0] offset:17408\n\t" \
    "v_mfma_f32_16x16x128_f8f6f4 v[24:27], v[80:87], v[56:63], 0\n\t" \
    "ds_read_b128 v[52:55], %[ab1] offset:8192\n\t" \
    "ds_read_b64_tr_b8 v[100:101], %[wb0] offset:24576\n\t" \
    "ds_read_b64_tr_b8 v[102:103], %[wb0] offset:25600\n\t" \
    "v_mfma_f32_16x16x128_f8f6f4 v[28:31], v[88:95], v[56:63], 0\n\t" \
    "ds_read_b128 v[56:59], %[ab0] offset:10240\n\t" \
    "ds_read_b64_tr_b8 v[104:105], %[wb1] offset:16384\n\t" \
    "ds_read_b64_tr_b8 v[106:107], %[wb1] offset:17408\n\t" \
    "s_waitcnt lgkmcnt(11)\n\t" \
    "v_mfma_f32_16x16x128_f8f6f4 v[32:35], v[80:87], v[64:71], 0\n\t" \
    "ds_read_b128 v[60:63], %[ab1] offset:10240\n\t" \
    "ds_read_b64_tr_b8 v[108:109], %[wb1] offset:24576\n\t" \
    "ds_read_b64_tr_b8 v[110:111], %[wb1] offset:25600\n\t" \
    "v_mfma_f32_16x16x128_f8f6f4 v[36:39], v[88:95], v[64:71], 0\n\t" \
    "ds_read_b128 v[64:67], %[ab0] offset:12288\n\t" \
    "s_waitcnt lgkmcnt(13)\n\t" \
    "v_mfma_f32_16x16x128_f8f6f4 v[40:43], v[80:87], v[72:79], 0\n\t" \
    "ds_read_b128 v[68:71], %[ab1] offset:12288\n\t" \
    "v_mfma_f32_16x16x128_f8f6f4 v[44:47], v[88:95], v[72:79], 0\n\t" \
    "ds_read_b128 v[72:75], %[ab0] offset:14336\n\t" \
    "s_waitcnt lgkmcnt(9)\n\t" \
    "v_mfma_f32_16x16x128_f8f6f4 v[16:19], v[96:103], v[48:55], v[16:19]\n\t" \
    "ds_read_b128 v[76:79], %[ab1] offset:14336\n\t" \
    "s_barrier\n\t" \
    "s_waitcnt lgkmcnt(4)\n\t" \
    "v_mfma_f32_16x16x128_f8f6f4 v[20:23], v[104:111], v[48:55], v[20:23]\n\t" \
    "ds_read_b128 v[48:51], %[ab0] offset:16384\n\t" \
    "ds_read_b64_tr_b8 v[112:113], %[wb0] offset:32768\n\t" \
    "ds_read_b64_tr_b8 v[114:115], %[wb0] offset:33792\n\t" \
    "v_mfma_f32_16x16x128_f8f6f4 v[24:27], v[96:103], v[56:63], v[24:27]\n\t" \
    "ds_read_b128 v[52:55], %[ab1] offset:16384\n\t" \
    "ds_read_b64_tr_b8 v[116:117], %[wb0] offset:40960\n\t" \
    "ds_read_b64_tr_b8 v[118:119], %[wb0] offset:41984\n\t" \
    "v_mfma_f32_16x16x128_f8f6f4 v[28:31], v[104:111], v[56:63], v[28:31]\n\t" \
    "ds_read_b128 v[56:59], %[ab0] offset:18432\n\t" \
    "ds_read_b64_tr_b8 v[120:121], %[wb1] offset:32768\n\t" \
    "ds_read_b64_tr_b8 v[122:123], %[wb1] offset:33792\n\t" \
    "s_waitcnt lgkmcnt(11)\n\t" \
    "v_mfma_f32_16x16x128_f8f6f4 v[32:35], v[96:103], v[64:71], v[32:35]\n\t" \
    "ds_read_b128 v[60:63], %[ab1] offset:18432\n\t" \
    "ds_read_b64_tr_b8 v[124:125], %[wb1] offset:40960\n\t" \
    "ds_read_b64_tr_b8 v[126:127], %[wb1] offset:41984\n\t" \
    "v_mfma_f32_16x16x128_f8f6f4 v[36:39], v[104:111], v[64:71], v[36:39]\n\t" \
    "ds_read_b128 v[64:67], %[ab0] offset:20480\n\t" \
    "s_waitcnt lgkmcnt(13)\n\t" \
    "v_mfma_f32_16x16x128_f8f6f4 v[40:43], v[96:103], v[72:79], v[40:43]\n\t" \
    "ds_read_b128 v[68:71], %[ab1] offset:20480\n\t" \
    "v_mfma_f32_16x16x128_f8f6f4 v[44:47], v[104:111], v[72:79], v[44:47]\n\t" \
    "ds_read_b128 v[72:75], %[ab0] offset:22528\n\t" \
    "s_waitcnt lgkmcnt(9)\n\t" \
    "v_mfma_f32_16x16x128_f8f6f4 v[16:19], v[112:119], v[48:55], v[16:19]\n\t" \
    "ds_read_b128 v[76:79], %[ab1] offset:22528\n\t" \
    "s_barrier\n\t" \
    "s_waitcnt lgkmcnt(4)\n\t" \
    "v_mfma_f32_16x16x128_f8f6f4 v[20:23], v[120:127], v[48:55], v[20:23]\n\t" \
    "ds_read_b128 v[48:51], %[ab0] offset:24576\n\t" \
    "ds_read_b64_tr_b8 v[128:129], %[wb0] offset:49152\n\t" \
    "ds_read_b64_tr_b8 v[130:131], %[wb0] offset:50176\n\t" \
    "v_mfma_f32_16x16x128_f8f6f4 v[24:27], v[112:119], v[56:63], v[24:27]\n\t" \
    "ds_read_b128 v[52:55], %[ab1] offset:24576\n\t" \
    "ds_read_b64_tr_b8 v[132:133], %[wb0] offset:57344\n\t" \
    "ds_read_b64_tr_b8 v[134:135], %[wb0] offset:58368\n\t" \
    "v_mfma_f32_16x16x128_f8f6f4 v[28:31], v[120:127], v[56:63], v[28:31]\n\t" \
    "ds_read_b128 v[56:59], %[ab0] offset:26624\n\t" \
    "ds_read_b64_tr_b8 v[136:137], %[wb1] offset:49152\n\t" \
    "ds_read_b64_tr_b8 v[138:139], %[wb1] offset:50176\n\t" \
    "s_waitcnt lgkmcnt(11)\n\t" \
    "v_mfma_f32_16x16x128_f8f6f4 v[32:35], v[112:119], v[64:71], v[32:35]\n\t" \
    "ds_read_b128 v[60:63], %[ab1] offset:26624\n\t" \
    "ds_read_b64_tr_b8 v[140:141], %[wb1] offset:57344\n\t" \
    "ds_read_b64_tr_b8 v[142:143], %[wb1] offset:58368\n\t" \
    "v_mfma_f32_16x16x128_f8f6f4 v[36:39], v[120:127], v[64:71], v[36:39]\n\t" \
    "ds_read_b128 v[64:67], %[ab0] offset:28672\n\t" \
    "s_waitcnt lgkmcnt(13)\n\t" \
    "v_mfma_f32_16x16x128_f8f6f4 v[40:43], v[112:119], v[72:79], v[40:43]\n\t" \
    "ds_read_b128 v[68:71], %[ab1] offset:28672\n\t" \
    "v_mfma_f32_16x16x128_f8f6f4 v[44:47], v[120:127], v[72:79], v[44:47]\n\t" \
    "ds_read_b128 v[72:75], %[ab0] offset:30720\n\t" \
    "s_waitcnt lgkmcnt(9)\n\t" \
    "v_mfma_f32_16x16x128_f8f6f4 v[16:19], v[128:135], v[48:55], v[16:19]\n\t" \
    "ds_read_b128 v[76:79], %[ab1] offset:30720\n\t" \
    "s_barrier\n\t" \
    "s_waitcnt lgkmcnt(4)\n\t" \
    "v_mfma_f32_16x16x128_f8f6f4 v[20:23], v[136:143], v[48:55], v[20:23]\n\t" \
    "ds_read_b64_tr_b8 v[144:145], %[wb0h] offset:0\n\t" \
    "v_mfma_f32_16x16x128_f8f6f4 v[24:27], v[128:135], v[56:63], v[24:27]\n\t" \
    "ds_read_b64_tr_b8 v[146:147], %[wb0h] offset:1024\n\t" \
    "v_mfma_f32_16x16x128_f8f6f4 v[28:31], v[136:143], v[56:63], v[28:31]\n\t" \
    "ds_read_b64_tr_b8 v[148:149], %[wb0h] offset:8192\n\t" \
    "s_waitcnt lgkmcnt(5)\n\t" \
    "v_mfma_f32_16x16x128_f8f6f4 v[32:35], v[128:135], v[64:71], v[32:35]\n\t" \
    "ds_read_b64_tr_b8 v[150:151], %[wb0h] offset:9216\n\t" \
    "v_mfma_f32_16x16x128_f8f6f4 v[36:39], v[136:143], v[64:71], v[36:39]\n\t" \
    "ds_read_b64_tr_b8 v[152:153], %[wb1h] offset:0\n\t" \
    "s_waitcnt lgkmcnt(5)\n\t" \
    "v_mfma_f32_16x16x128_f8f6f4 v[40:43], v[128:135], v[72:79], v[40:43]\n\t" \
    "ds_read_b64_tr_b8 v[154:155], %[wb1h] offset:1024\n\t" \
    "v_mfma_f32_16x16x128_f8f6f4 v[44:47], v[136:143], v[72:79], v[44:47]\n\t" \
    "ds_read_b64_tr_b8 v[156:157], %[wb1h] offset:8192\n\t" \
    "ds_read_b64_tr_b8 v[158:159], %[wb1h] offset:9216\n\t" \
    "s_barrier\n\t" \
    "ds_read_b64_tr_b8 v[160:161], %[wb0h] offset:16384\n\t" \
    "ds_read_b64_tr_b8 v[162:163], %[wb0h] offset:17408\n\t" \
    "ds_read_b64_tr_b8 v[164:165], %[wb0h] offset:24576\n\t" \
    "ds_read_b64_tr_b8 v[166:167], %[wb0h] offset:25600\n\t" \
    "ds_read_b64_tr_b8 v[168:169], %[wb1h] offset:16384\n\t" \
    "ds_read_b64_tr_b8 v[170:171], %[wb1h] offset:17408\n\t" \
    "ds_read_b64_tr_b8 v[172:173], %[wb1h] offset:24576\n\t" \
    "ds_read_b64_tr_b8 v[174:175], %[wb1h] offset:25600\n\t" \
    "s_waitcnt lgkmcnt(8)\n\t" \
    "s_barrier\n\t" \
    "ds_read_b64_tr_b8 v[176:177], %[wb0h] offset:32768\n\t" \
    "ds_read_b64_tr_b8 v[178:179], %[wb0h] offset:33792\n\t" \
    "ds_read_b64_tr_b8 v[180:181], %[wb0h] offset:40960\n\t" \
    "ds_read_b64_tr_b8 v[182:183], %[wb0h] offset:41984\n\t" \
    "ds_read_b64_tr_b8 v[184:185], %[wb1h] offset:32768\n\t" \
    "ds_read_b64_tr_b8 v[186:187], %[wb1h] offset:33792\n\t" \
    "ds_read_b64_tr_b8 v[188:189], %[wb1h] offset:40960\n\t" \
    "ds_read_b64_tr_b8 v[190:191], %[wb1h] offset:41984\n\t" \
    "s_waitcnt lgkmcnt(8)\n\t" \
    "s_barrier\n\t" \
    "ds_read_b64_tr_b8 v[192:193], %[wb0] offset:0\n\t" \
    "ds_read_b64_tr_b8 v[194:195], %[wb0] offset:1024\n\t" \
    "ds_read_b64_tr_b8 v[196:197], %[wb0] offset:8192\n\t" \
    "ds_read_b64_tr_b8 v[198:199], %[wb0] offset:9216\n\t" \
    "ds_read_b64_tr_b8 v[200:201], %[wb1] offset:0\n\t" \
    "ds_read_b64_tr_b8 v[202:203], %[wb1] offset:1024\n\t" \
    "ds_read_b64_tr_b8 v[204:205], %[wb1] offset:8192\n\t" \
    "ds_read_b64_tr_b8 v[206:207], %[wb1] offset:9216\n\t" \
    "s_waitcnt lgkmcnt(8)\n\t" \
    "s_barrier\n\t" \
    "ds_read_b64_tr_b8 v[208:209], %[wb0] offset:16384\n\t" \
    "ds_read_b64_tr_b8 v[210:211], %[wb0] offset:17408\n\t" \
    "ds_read_b64_tr_b8 v[212:213], %[wb0] offset:24576\n\t" \
    "ds_read_b64_tr_b8 v[214:215], %[wb0] offset:25600\n\t" \
    "ds_read_b64_tr_b8 v[216:217], %[wb1] offset:16384\n\t" \
    "ds_read_b64_tr_b8 v[218:219], %[wb1] offset:17408\n\t" \
    "ds_read_b64_tr_b8 v[220:221], %[wb1] offset:24576\n\t" \
    "ds_read_b64_tr_b8 v[222:223], %[wb1] offset:25600\n\t" \
    "s_waitcnt lgkmcnt(8)\n\t" \
    "s_barrier\n\t" \
    "ds_read_b64_tr_b8 v[224:225], %[wb0] offset:32768\n\t" \
    "ds_read_b64_tr_b8 v[226:227], %[wb0] offset:33792\n\t" \
    "ds_read_b64_tr_b8 v[228:229], %[wb0] offset:40960\n\t" \
    "ds_read_b64_tr_b8 v[230:231], %[wb0] offset:41984\n\t" \
    "ds_read_b64_tr_b8 v[232:233], %[wb1] offset:32768\n\t" \
    "ds_read_b64_tr_b8 v[234:235], %[wb1] offset:33792\n\t" \
    "ds_read_b64_tr_b8 v[236:237], %[wb1] offset:40960\n\t" \
    "ds_read_b64_tr_b8 v[238:239], %[wb1] offset:41984\n\t" \
    "s_waitcnt lgkmcnt(8)\n\t" \
    "s_barrier\n\t" \
    "ds_read_b64_tr_b8 v[240:241], %[wb0] offset:49152\n\t" \
    "ds_read_b64_tr_b8 v[242:243], %[wb0] offset:50176\n\t" \
    "ds_read_b64_tr_b8 v[244:245], %[wb0] offset:57344\n\t" \
    "ds_read_b64_tr_b8 v[246:247], %[wb0] offset:58368\n\t" \
    "ds_read_b64_tr_b8 v[248:249], %[wb1] offset:49152\n\t" \
    "ds_read_b64_tr_b8 v[250:251], %[wb1] offset:50176\n\t" \
    "ds_read_b64_tr_b8 v[252:253], %[wb1] offset:57344\n\t" \
    "ds_read_b64_tr_b8 v[254:255], %[wb1] offset:58368\n\t" \
    "s_waitcnt lgkmcnt(8)\n\t" \
    "s_barrier\n\t" \
    "ds_read_b64_tr_b8 v[80:81], %[wb0h] offset:0\n\t" \
    "ds_read_b64_tr_b8 v[82:83], %[wb0h] offset:1024\n\t" \
    "ds_read_b64_tr_b8 v[84:85], %[wb0h] offset:8192\n\t" \
    "ds_read_b64_tr_b8 v[86:87], %[wb0h] offset:9216\n\t" \
    "ds_read_b64_tr_b8 v[88:89], %[wb1h] offset:0\n\t" \
    "ds_read_b64_tr_b8 v[90:91], %[wb1h] offset:1024\n\t" \
    "ds_read_b64_tr_b8 v[92:93], %[wb1h] offset:8192\n\t" \
    "ds_read_b64_tr_b8 v[94:95], %[wb1h] offset:9216\n\t" \
    "s_waitcnt lgkmcnt(8)\n\t" \
    "s_barrier\n\t" \
    "ds_read_b64_tr_b8 v[96:97], %[wb0h] offset:16384\n\t" \
    "ds_read_b64_tr_b8 v[98:99], %[wb0h] offset:17408\n\t" \
    "ds_read_b64_tr_b8 v[100:101], %[wb0h] offset:24576\n\t" \
    "ds_read_b64_tr_b8 v[102:103], %[wb0h] offset:25600\n\t" \
    "ds_read_b64_tr_b8 v[104:105], %[wb1h] offset:16384\n\t" \
    "ds_read_b64_tr_b8 v[106:107], %[wb1h] offset:17408\n\t" \
    "ds_read_b64_tr_b8 v[108:109], %[wb1h] offset:24576\n\t" \
    "ds_read_b64_tr_b8 v[110:111], %[wb1h] offset:25600\n\t" \
    "s_waitcnt lgkmcnt(8)\n\t" \
    "s_barrier\n\t" \
    "ds_read_b64_tr_b8 v[112:113], %[wb0h] offset:32768\n\t" \
    "ds_read_b64_tr_b8 v[114:115], %[wb0h] offset:33792\n\t" \
    "ds_read_b64_tr_b8 v[116:117], %[wb0h] offset:40960\n\t" \
    "ds_read_b64_tr_b8 v[118:119], %[wb0h] offset:41984\n\t" \
    "ds_read_b64_tr_b8 v[120:121], %[wb1h] offset:32768\n\t" \
    "ds_read_b64_tr_b8 v[122:123], %[wb1h] offset:33792\n\t" \
    "ds_read_b64_tr_b8 v[124:125], %[wb1h] offset:40960\n\t" \
    "ds_read_b64_tr_b8 v[126:127], %[wb1h] offset:41984\n\t" \
    "s_waitcnt lgkmcnt(8)\n\t" \
    "s_barrier\n\t" \
    "ds_read_b64_tr_b8 v[128:129], %[wb0] offset:0\n\t" \
    "ds_read_b64_tr_b8 v[130:131], %[wb0] offset:1024\n\t" \
    "ds_read_b64_tr_b8 v[132:133], %[wb0] offset:8192\n\t" \
    "ds_read_b64_tr_b8 v[134:135], %[wb0] offset:9216\n\t" \
    "ds_read_b64_tr_b8 v[136:137], %[wb1] offset:0\n\t" \
    "ds_read_b64_tr_b8 v[138:139], %[wb1] offset:1024\n\t" \
    "ds_read_b64_tr_b8 v[140:141], %[wb1] offset:8192\n\t" \
    "ds_read_b64_tr_b8 v[142:143], %[wb1] offset:9216\n\t" \
    "s_waitcnt lgkmcnt(8)\n\t" \
    "s_barrier\n\t" \
    "s_waitcnt lgkmcnt(0)\n\t" \
    "s_barrier\n\t" \
    "ds_read_b128 v[48:51], %[ab0] offset:32768\n\t" \
    "ds_read_b128 v[52:55], %[ab1] offset:32768\n\t" \
    "ds_read_b128 v[56:59], %[ab0] offset:34816\n\t" \
    "ds_read_b128 v[60:63], %[ab1] offset:34816\n\t" \
    "ds_read_b128 v[64:67], %[ab0] offset:36864\n\t" \
    "ds_read_b128 v[68:71], %[ab1] offset:36864\n\t" \
    "ds_read_b128 v[72:75], %[ab0] offset:38912\n\t" \
    "ds_read_b128 v[76:79], %[ab1] offset:38912\n\t" \
    "s_waitcnt lgkmcnt(6)\n\t" \
    "v_mfma_f32_16x16x128_f8f6f4 v[16:19], v[144:151], v[48:55], v[16:19]\n\t" \
    "s_barrier\n\t" \
    "v_mfma_f32_16x16x128_f8f6f4 v[20:23], v[152:159], v[48:55], v[20:23]\n\t" \
    "ds_read_b128 v[48:51], %[ab0] offset:0\n\t" \
    "s_waitcnt lgkmcnt(5)\n\t" \
    "v_mfma_f32_16x16x128_f8f6f4 v[24:27], v[144:151], v[56:63], v[24:27]\n\t" \
    "ds_read_b128 v[52:55], %[ab1] offset:0\n\t" \
    "v_mfma_f32_16x16x128_f8f6f4 v[28:31], v[152:159], v[56:63], v[28:31]\n\t" \
    "ds_read_b128 v[56:59], %[ab0] offset:2048\n\t" \
    "s_waitcnt lgkmcnt(5)\n\t" \
    "v_mfma_f32_16x16x128_f8f6f4 v[32:35], v[144:151], v[64:71], v[32:35]\n\t" \
    "ds_read_b128 v[60:63], %[ab1] offset:2048\n\t" \
    "v_mfma_f32_16x16x128_f8f6f4 v[36:39], v[152:159], v[64:71], v[36:39]\n\t" \
    "ds_read_b128 v[64:67], %[ab0] offset:4096\n\t" \
    "s_waitcnt lgkmcnt(5)\n\t" \
    "v_mfma_f32_16x16x128_f8f6f4 v[40:43], v[144:151], v[72:79], v[40:43]\n\t" \
    "ds_read_b128 v[68:71], %[ab1] offset:4096\n\t" \
    "v_mfma_f32_16x16x128_f8f6f4 v[44:47], v[152:159], v[72:79], v[44:47]\n\t" \
    "ds_read_b128 v[72:75], %[ab0] offset:6144\n\t" \
    "s_waitcnt lgkmcnt(5)\n\t" \
    "v_mfma_f32_16x16x128_f8f6f4 v[16:19], v[160:167], v[48:55], v[16:19]\n\t" \
    "ds_read_b128 v[76:79], %[ab1] offset:6144\n\t" \
    "s_barrier\n\t" \
    "v_mfma_f32_16x16x128_f8f6f4 v[20:23], v[168:175], v[48:55], v[20:23]\n\t" \
    "ds_read_b128 v[48:51], %[ab0] offset:8192\n\t" \
    "ds_read_b64_tr_b8 v[144:145], %[wb0] offset:16384\n\t" \
    "s_waitcnt lgkmcnt(6)\n\t" \
    "v_mfma_f32_16x16x128_f8f6f4 v[24:27], v[160:167], v[56:63], v[24:27]\n\t" \
    "ds_read_b128 v[52:55], %[ab1] offset:8192\n\t" \
    "ds_read_b64_tr_b8 v[146:147], %[wb0] offset:17408\n\t" \
    "v_mfma_f32_16x16x128_f8f6f4 v[28:31], v[168:175], v[56:63], v[28:31]\n\t" \
    "ds_read_b128 v[56:59], %[ab0] offset:10240\n\t" \
    "ds_read_b64_tr_b8 v[148:149], %[wb0] offset:24576\n\t" \
    "s_waitcnt lgkmcnt(8)\n\t" \
    "v_mfma_f32_16x16x128_f8f6f4 v[32:35], v[160:167], v[64:71], v[32:35]\n\t" \
    "ds_read_b128 v[60:63], %[ab1] offset:10240\n\t" \
    "ds_read_b64_tr_b8 v[150:151], %[wb0] offset:25600\n\t" \
    "v_mfma_f32_16x16x128_f8f6f4 v[36:39], v[168:175], v[64:71], v[36:39]\n\t" \
    "ds_read_b128 v[64:67], %[ab0] offset:12288\n\t" \
    "ds_read_b64_tr_b8 v[152:153], %[wb1] offset:16384\n\t" \
    "s_waitcnt lgkmcnt(10)\n\t" \
    "v_mfma_f32_16x16x128_f8f6f4 v[40:43], v[160:167], v[72:79], v[40:43]\n\t" \
    "ds_read_b128 v[68:71], %[ab1] offset:12288\n\t" \
    "ds_read_b64_tr_b8 v[154:155], %[wb1] offset:17408\n\t" \
    "v_mfma_f32_16x16x128_f8f6f4 v[44:47], v[168:175], v[72:79], v[44:47]\n\t" \
    "ds_read_b128 v[72:75], %[ab0] offset:14336\n\t" \
    "ds_read_b64_tr_b8 v[156:157], %[wb1] offset:24576\n\t" \
    "s_waitcnt lgkmcnt(11)\n\t" \
    "v_mfma_f32_16x16x128_f8f6f4 v[16:19], v[176:183], v[48:55], v[16:19]\n\t" \
    "ds_read_b128 v[76:79], %[ab1] offset:14336\n\t" \
    "ds_read_b64_tr_b8 v[158:159], %[wb1] offset:25600\n\t" \
    "s_barrier\n\t" \
    "v_mfma_f32_16x16x128_f8f6f4 v[20:23], v[184:191], v[48:55], v[20:23]\n\t" \
    "ds_read_b128 v[48:51], %[ab0] offset:16384\n\t" \
    "ds_read_b64_tr_b8 v[160:161], %[wb0] offset:32768\n\t" \
    "s_waitcnt lgkmcnt(11)\n\t" \
    "v_mfma_f32_16x16x128_f8f6f4 v[24:27], v[176:183], v[56:63], v[24:27]\n\t" \
    "ds_read_b128 v[52:55], %[ab1] offset:16384\n\t" \
    "ds_read_b64_tr_b8 v[162:163], %[wb0] offset:33792\n\t" \
    "v_mfma_f32_16x16x128_f8f6f4 v[28:31], v[184:191], v[56:63], v[28:31]\n\t" \
    "ds_read_b128 v[56:59], %[ab0] offset:18432\n\t" \
    "ds_read_b64_tr_b8 v[164:165], %[wb0] offset:40960\n\t" \
    "s_waitcnt lgkmcnt(11)\n\t" \
    "v_mfma_f32_16x16x128_f8f6f4 v[32:35], v[176:183], v[64:71], v[32:35]\n\t" \
    "ds_read_b128 v[60:63], %[ab1] offset:18432\n\t" \
    "ds_read_b64_tr_b8 v[166:167], %[wb0] offset:41984\n\t" \
    "v_mfma_f32_16x16x128_f8f6f4 v[36:39], v[184:191], v[64:71], v[36:39]\n\t" \
    "ds_read_b128 v[64:67], %[ab0] offset:20480\n\t" \
    "ds_read_b64_tr_b8 v[168:169], %[wb1] offset:32768\n\t" \
    "s_waitcnt lgkmcnt(11)\n\t" \
    "v_mfma_f32_16x16x128_f8f6f4 v[40:43], v[176:183], v[72:79], v[40:43]\n\t" \
    "ds_read_b128 v[68:71], %[ab1] offset:20480\n\t" \
    "ds_read_b64_tr_b8 v[170:171], %[wb1] offset:33792\n\t" \
    "v_mfma_f32_16x16x128_f8f6f4 v[44:47], v[184:191], v[72:79], v[44:47]\n\t" \
    "ds_read_b128 v[72:75], %[ab0] offset:22528\n\t" \
    "ds_read_b64_tr_b8 v[172:173], %[wb1] offset:40960\n\t" \
    "s_waitcnt lgkmcnt(11)\n\t" \
    "v_mfma_f32_16x16x128_f8f6f4 v[16:19], v[192:199], v[48:55], v[16:19]\n\t" \
    "ds_read_b128 v[76:79], %[ab1] offset:22528\n\t" \
    "ds_read_b64_tr_b8 v[174:175], %[wb1] offset:41984\n\t" \
    "s_barrier\n\t" \
    "v_mfma_f32_16x16x128_f8f6f4 v[20:23], v[200:207], v[48:55], v[20:23]\n\t" \
    "ds_read_b128 v[48:51], %[ab0] offset:24576\n\t" \
    "ds_read_b64_tr_b8 v[176:177], %[wb0] offset:49152\n\t" \
    "s_waitcnt lgkmcnt(11)\n\t" \
    "v_mfma_f32_16x16x128_f8f6f4 v[24:27], v[192:199], v[56:63], v[24:27]\n\t" \
    "ds_read_b128 v[52:55], %[ab1] offset:24576\n\t" \
    "ds_read_b64_tr_b8 v[178:179], %[wb0] offset:50176\n\t" \
    "v_mfma_f32_16x16x128_f8f6f4 v[28:31], v[200:207], v[56:63], v[28:31]\n\t" \
    "ds_read_b128 v[56:59], %[ab0] offset:26624\n\t" \
    "ds_read_b64_tr_b8 v[180:181], %[wb0] offset:57344\n\t" \
    "s_waitcnt lgkmcnt(11)\n\t" \
    "v_mfma_f32_16x16x128_f8f6f4 v[32:35], v[192:199], v[64:71], v[32:35]\n\t" \
    "ds_read_b128 v[60:63], %[ab1] offset:26624\n\t" \
    "ds_read_b64_tr_b8 v[182:183], %[wb0] offset:58368\n\t" \
    "v_mfma_f32_16x16x128_f8f6f4 v[36:39], v[200:207], v[64:71], v[36:39]\n\t" \
    "ds_read_b128 v[64:67], %[ab0] offset:28672\n\t" \
    "ds_read_b64_tr_b8 v[184:185], %[wb1] offset:49152\n\t" \
    "s_waitcnt lgkmcnt(11)\n\t" \
    "v_mfma_f32_16x16x128_f8f6f4 v[40:43], v[192:199], v[72:79], v[40:43]\n\t" \
    "ds_read_b128 v[68:71], %[ab1] offset:28672\n\t" \
    "ds_read_b64_tr_b8 v[186:187], %[wb1] offset:50176\n\t" \
    "v_mfma_f32_16x16x128_f8f6f4 v[44:47], v[200:207], v[72:79], v[44:47]\n\t" \
    "ds_read_b128 v[72:75], %[ab0] offset:30720\n\t" \
    "ds_read_b64_tr_b8 v[188:189], %[wb1] offset:57344\n\t" \
    "s_waitcnt lgkmcnt(11)\n\t" \
    "v_mfma_f32_16x16x128_f8f6f4 v[16:19], v[208:215], v[48:55], v[16:19]\n\t" \
    "ds_read_b128 v[76:79], %[ab1] offset:30720\n\t" \
    "ds_read_b64_tr_b8 v[190:191], %[wb1] offset:58368\n\t" \
    "s_barrier\n\t" \
    "v_mfma_f32_16x16x128_f8f6f4 v[20:23], v[216:223], v[48:55], v[20:23]\n\t" \
    "ds_read_b128 v[48:51], %[ab0] offset:32768\n\t" \
    "ds_read_b64_tr_b8 v[192:193], %[wb0h] offset:0\n\t" \
    "s_waitcnt lgkmcnt(11)\n\t" \
    "v_mfma_f32_16x16x128_f8f6f4 v[24:27], v[208:215], v[56:63], v[24:27]\n\t" \
    "ds_read_b128 v[52:55], %[ab1] offset:32768\n\t" \
    "ds_read_b64_tr_b8 v[194:195], %[wb0h] offset:1024\n\t" \
    "v_mfma_f32_16x16x128_f8f6f4 v[28:31], v[216:223], v[56:63], v[28:31]\n\t" \
    "ds_read_b128 v[56:59], %[ab0] offset:34816\n\t" \
    "ds_read_b64_tr_b8 v[196:197], %[wb0h] offset:8192\n\t" \
    "s_waitcnt lgkmcnt(11)\n\t" \
    "v_mfma_f32_16x16x128_f8f6f4 v[32:35], v[208:215], v[64:71], v[32:35]\n\t" \
    "ds_read_b128 v[60:63], %[ab1] offset:34816\n\t" \
    "ds_read_b64_tr_b8 v[198:199], %[wb0h] offset:9216\n\t" \
    "v_mfma_f32_16x16x128_f8f6f4 v[36:39], v[216:223], v[64:71], v[36:39]\n\t" \
    "ds_read_b128 v[64:67], %[ab0] offset:36864\n\t" \
    "ds_read_b64_tr_b8 v[200:201], %[wb1h] offset:0\n\t" \
    "s_waitcnt lgkmcnt(11)\n\t" \
    "v_mfma_f32_16x16x128_f8f6f4 v[40:43], v[208:215], v[72:79], v[40:43]\n\t" \
    "ds_read_b128 v[68:71], %[ab1] offset:36864\n\t" \
    "ds_read_b64_tr_b8 v[202:203], %[wb1h] offset:1024\n\t" \
    "v_mfma_f32_16x16x128_f8f6f4 v[44:47], v[216:223], v[72:79], v[44:47]\n\t" \
    "ds_read_b128 v[72:75], %[ab0] offset:38912\n\t" \
    "ds_read_b64_tr_b8 v[204:205], %[wb1h] offset:8192\n\t" \
    "s_waitcnt lgkmcnt(11)\n\t" \
    "v_mfma_f32_16x16x128_f8f6f4 v[16:19], v[224:231], v[48:55], v[16:19]\n\t" \
    "ds_read_b128 v[76:79], %[ab1] offset:38912\n\t" \
    "ds_read_b64_tr_b8 v[206:207], %[wb1h] offset:9216\n\t" \
    "s_barrier\n\t" \
    "v_mfma_f32_16x16x128_f8f6f4 v[20:23], v[232:239], v[48:55], v[20:23]\n\t" \
    "ds_read_b128 v[48:51], %[ab0] offset:0\n\t" \
    "ds_read_b64_tr_b8 v[208:209], %[wb0h] offset:16384\n\t" \
    "s_waitcnt lgkmcnt(11)\n\t" \
    "v_mfma_f32_16x16x128_f8f6f4 v[24:27], v[224:231], v[56:63], v[24:27]\n\t" \
    "ds_read_b128 v[52:55], %[ab1] offset:0\n\t" \
    "ds_read_b64_tr_b8 v[210:211], %[wb0h] offset:17408\n\t" \
    "v_mfma_f32_16x16x128_f8f6f4 v[28:31], v[232:239], v[56:63], v[28:31]\n\t" \
    "ds_read_b128 v[56:59], %[ab0] offset:2048\n\t" \
    "ds_read_b64_tr_b8 v[212:213], %[wb0h] offset:24576\n\t" \
    "s_waitcnt lgkmcnt(11)\n\t" \
    "v_mfma_f32_16x16x128_f8f6f4 v[32:35], v[224:231], v[64:71], v[32:35]\n\t" \
    "ds_read_b128 v[60:63], %[ab1] offset:2048\n\t" \
    "ds_read_b64_tr_b8 v[214:215], %[wb0h] offset:25600\n\t" \
    "v_mfma_f32_16x16x128_f8f6f4 v[36:39], v[232:239], v[64:71], v[36:39]\n\t" \
    "ds_read_b128 v[64:67], %[ab0] offset:4096\n\t" \
    "ds_read_b64_tr_b8 v[216:217], %[wb1h] offset:16384\n\t" \
    "s_waitcnt lgkmcnt(11)\n\t" \
    "v_mfma_f32_16x16x128_f8f6f4 v[40:43], v[224:231], v[72:79], v[40:43]\n\t" \
    "ds_read_b128 v[68:71], %[ab1] offset:4096\n\t" \
    "ds_read_b64_tr_b8 v[218:219], %[wb1h] offset:17408\n\t" \
    "v_mfma_f32_16x16x128_f8f6f4 v[44:47], v[232:239], v[72:79], v[44:47]\n\t" \
    "ds_read_b128 v[72:75], %[ab0] offset:6144\n\t" \
    "ds_read_b64_tr_b8 v[220:221], %[wb1h] offset:24576\n\t" \
    "s_waitcnt lgkmcnt(11)\n\t" \
    "v_mfma_f32_16x16x128_f8f6f4 v[16:19], v[240:247], v[48:55], v[16:19]\n\t" \
    "ds_read_b128 v[76:79], %[ab1] offset:6144\n\t" \
    "ds_read_b64_tr_b8 v[222:223], %[wb1h] offset:25600\n\t" \
    "s_barrier\n\t" \
    "v_mfma_f32_16x16x128_f8f6f4 v[20:23], v[248:255], v[48:55], v[20:23]\n\t" \
    "ds_read_b128 v[48:51], %[ab0] offset:8192\n\t" \
    "ds_read_b64_tr_b8 v[224:225], %[wb0h] offset:32768\n\t" \
    "s_waitcnt lgkmcnt(11)\n\t" \
    "v_mfma_f32_16x16x128_f8f6f4 v[24:27], v[240:247], v[56:63], v[24:27]\n\t" \
    "ds_read_b128 v[52:55], %[ab1] offset:8192\n\t" \
    "ds_read_b64_tr_b8 v[226:227], %[wb0h] offset:33792\n\t" \
    "v_mfma_f32_16x16x128_f8f6f4 v[28:31], v[248:255], v[56:63], v[28:31]\n\t" \
    "ds_read_b128 v[56:59], %[ab0] offset:10240\n\t" \
    "ds_read_b64_tr_b8 v[228:229], %[wb0h] offset:40960\n\t" \
    "s_waitcnt lgkmcnt(11)\n\t" \
    "v_mfma_f32_16x16x128_f8f6f4 v[32:35], v[240:247], v[64:71], v[32:35]\n\t" \
    "ds_read_b128 v[60:63], %[ab1] offset:10240\n\t" \
    "ds_read_b64_tr_b8 v[230:231], %[wb0h] offset:41984\n\t" \
    "v_mfma_f32_16x16x128_f8f6f4 v[36:39], v[248:255], v[64:71], v[36:39]\n\t" \
    "ds_read_b128 v[64:67], %[ab0] offset:12288\n\t" \
    "ds_read_b64_tr_b8 v[232:233], %[wb1h] offset:32768\n\t" \
    "s_waitcnt lgkmcnt(11)\n\t" \
    "v_mfma_f32_16x16x128_f8f6f4 v[40:43], v[240:247], v[72:79], v[40:43]\n\t" \
    "ds_read_b128 v[68:71], %[ab1] offset:12288\n\t" \
    "ds_read_b64_tr_b8 v[234:235], %[wb1h] offset:33792\n\t" \
    "v_mfma_f32_16x16x128_f8f6f4 v[44:47], v[248:255], v[72:79], v[44:47]\n\t" \
    "ds_read_b128 v[72:75], %[ab0] offset:14336\n\t" \
    "ds_read_b64_tr_b8 v[236:237], %[wb1h] offset:40960\n\t" \
    "s_waitcnt lgkmcnt(11)\n\t" \
    "v_mfma_f32_16x16x128_f8f6f4 v[16:19], v[80:87], v[48:55], v[16:19]\n\t" \
    "ds_read_b128 v[76:79], %[ab1] offset:14336\n\t" \
    "ds_read_b64_tr_b8 v[238:239], %[wb1h] offset:41984\n\t" \
    "s_barrier\n\t" \
    "v_mfma_f32_16x16x128_f8f6f4 v[20:23], v[88:95], v[48:55], v[20:23]\n\t" \
    "ds_read_b128 v[48:51], %[ab0] offset:16384\n\t" \
    "ds_read_b64_tr_b8 v[240:241], %[wb0] offset:0\n\t" \
    "s_waitcnt lgkmcnt(11)\n\t" \
    "v_mfma_f32_16x16x128_f8f6f4 v[24:27], v[80:87], v[56:63], v[24:27]\n\t" \
    "ds_read_b128 v[52:55], %[ab1] offset:16384\n\t" \
    "ds_read_b64_tr_b8 v[242:243], %[wb0] offset:1024\n\t" \
    "v_mfma_f32_16x16x128_f8f6f4 v[28:31], v[88:95], v[56:63], v[28:31]\n\t" \
    "ds_read_b128 v[56:59], %[ab0] offset:18432\n\t" \
    "ds_read_b64_tr_b8 v[244:245], %[wb0] offset:8192\n\t" \
    "s_waitcnt lgkmcnt(11)\n\t" \
    "v_mfma_f32_16x16x128_f8f6f4 v[32:35], v[80:87], v[64:71], v[32:35]\n\t" \
    "ds_read_b128 v[60:63], %[ab1] offset:18432\n\t" \
    "ds_read_b64_tr_b8 v[246:247], %[wb0] offset:9216\n\t" \
    "v_mfma_f32_16x16x128_f8f6f4 v[36:39], v[88:95], v[64:71], v[36:39]\n\t" \
    "ds_read_b128 v[64:67], %[ab0] offset:20480\n\t" \
    "ds_read_b64_tr_b8 v[248:249], %[wb1] offset:0\n\t" \
    "s_waitcnt lgkmcnt(11)\n\t" \
    "v_mfma_f32_16x16x128_f8f6f4 v[40:43], v[80:87], v[72:79], v[40:43]\n\t" \
    "ds_read_b128 v[68:71], %[ab1] offset:20480\n\t" \
    "ds_read_b64_tr_b8 v[250:251], %[wb1] offset:1024\n\t" \
    "v_mfma_f32_16x16x128_f8f6f4 v[44:47], v[88:95], v[72:79], v[44:47]\n\t" \
    "ds_read_b128 v[72:75], %[ab0] offset:22528\n\t" \
    "ds_read_b64_tr_b8 v[252:253], %[wb1] offset:8192\n\t" \
    "s_waitcnt lgkmcnt(11)\n\t" \
    "v_mfma_f32_16x16x128_f8f6f4 v[16:19], v[96:103], v[48:55], v[16:19]\n\t" \
    "ds_read_b128 v[76:79], %[ab1] offset:22528\n\t" \
    "ds_read_b64_tr_b8 v[254:255], %[wb1] offset:9216\n\t" \
    "s_barrier\n\t" \
    "v_mfma_f32_16x16x128_f8f6f4 v[20:23], v[104:111], v[48:55], v[20:23]\n\t" \
    "ds_read_b128 v[48:51], %[ab0] offset:24576\n\t" \
    "ds_read_b64_tr_b8 v[80:81], %[wb0] offset:16384\n\t" \
    "s_waitcnt lgkmcnt(11)\n\t" \
    "v_mfma_f32_16x16x128_f8f6f4 v[24:27], v[96:103], v[56:63], v[24:27]\n\t" \
    "ds_read_b128 v[52:55], %[ab1] offset:24576\n\t" \
    "ds_read_b64_tr_b8 v[82:83], %[wb0] offset:17408\n\t" \
    "v_mfma_f32_16x16x128_f8f6f4 v[28:31], v[104:111], v[56:63], v[28:31]\n\t" \
    "ds_read_b128 v[56:59], %[ab0] offset:26624\n\t" \
    "ds_read_b64_tr_b8 v[84:85], %[wb0] offset:24576\n\t" \
    "s_waitcnt lgkmcnt(11)\n\t" \
    "v_mfma_f32_16x16x128_f8f6f4 v[32:35], v[96:103], v[64:71], v[32:35]\n\t" \
    "ds_read_b128 v[60:63], %[ab1] offset:26624\n\t" \
    "ds_read_b64_tr_b8 v[86:87], %[wb0] offset:25600\n\t" \
    "v_mfma_f32_16x16x128_f8f6f4 v[36:39], v[104:111], v[64:71], v[36:39]\n\t" \
    "ds_read_b128 v[64:67], %[ab0] offset:28672\n\t" \
    "ds_read_b64_tr_b8 v[88:89], %[wb1] offset:16384\n\t" \
    "s_waitcnt lgkmcnt(11)\n\t" \
    "v_mfma_f32_16x16x128_f8f6f4 v[40:43], v[96:103], v[72:79], v[40:43]\n\t" \
    "ds_read_b128 v[68:71], %[ab1] offset:28672\n\t" \
    "ds_read_b64_tr_b8 v[90:91], %[wb1] offset:17408\n\t" \
    "v_mfma_f32_16x16x128_f8f6f4 v[44:47], v[104:111], v[72:79], v[44:47]\n\t" \
    "ds_read_b128 v[72:75], %[ab0] offset:30720\n\t" \
    "ds_read_b64_tr_b8 v[92:93], %[wb1] offset:24576\n\t" \
    "s_waitcnt lgkmcnt(11)\n\t" \
    "v_mfma_f32_16x16x128_f8f6f4 v[16:19], v[112:119], v[48:55], v[16:19]\n\t" \
    "ds_read_b128 v[76:79], %[ab1] offset:30720\n\t" \
    "ds_read_b64_tr_b8 v[94:95], %[wb1] offset:25600\n\t" \
    "s_barrier\n\t" \
    "v_mfma_f32_16x16x128_f8f6f4 v[20:23], v[120:127], v[48:55], v[20:23]\n\t" \
    "ds_read_b128 v[48:51], %[ab0] offset:32768\n\t" \
    "ds_read_b64_tr_b8 v[96:97], %[wb0] offset:32768\n\t" \
    "s_waitcnt lgkmcnt(11)\n\t" \
    "v_mfma_f32_16x16x128_f8f6f4 v[24:27], v[112:119], v[56:63], v[24:27]\n\t" \
    "ds_read_b128 v[52:55], %[ab1] offset:32768\n\t" \
    "ds_read_b64_tr_b8 v[98:99], %[wb0] offset:33792\n\t" \
    "v_mfma_f32_16x16x128_f8f6f4 v[28:31], v[120:127], v[56:63], v[28:31]\n\t" \
    "ds_read_b128 v[56:59], %[ab0] offset:34816\n\t" \
    "ds_read_b64_tr_b8 v[100:101], %[wb0] offset:40960\n\t" \
    "s_waitcnt lgkmcnt(11)\n\t" \
    "v_mfma_f32_16x16x128_f8f6f4 v[32:35], v[112:119], v[64:71], v[32:35]\n\t" \
    "ds_read_b128 v[60:63], %[ab1] offset:34816\n\t" \
    "ds_read_b64_tr_b8 v[102:103], %[wb0] offset:41984\n\t" \
    "v_mfma_f32_16x16x128_f8f6f4 v[36:39], v[120:127], v[64:71], v[36:39]\n\t" \
    "ds_read_b128 v[64:67], %[ab0] offset:36864\n\t" \
    "ds_read_b64_tr_b8 v[104:105], %[wb1] offset:32768\n\t" \
    "s_waitcnt lgkmcnt(11)\n\t" \
    "v_mfma_f32_16x16x128_f8f6f4 v[40:43], v[112:119], v[72:79], v[40:43]\n\t" \
    "ds_read_b128 v[68:71], %[ab1] offset:36864\n\t" \
    "ds_read_b64_tr_b8 v[106:107], %[wb1] offset:33792\n\t" \
    "v_mfma_f32_16x16x128_f8f6f4 v[44:47], v[120:127], v[72:79], v[44:47]\n\t" \
    "ds_read_b128 v[72:75], %[ab0] offset:38912\n\t" \
    "ds_read_b64_tr_b8 v[108:109], %[wb1] offset:40960\n\t" \
    "s_waitcnt lgkmcnt(11)\n\t" \
    "v_mfma_f32_16x16x128_f8f6f4 v[16:19], v[128:135], v[48:55], v[16:19]\n\t" \
    "ds_read_b128 v[76:79], %[ab1] offset:38912\n\t" \
    "ds_read_b64_tr_b8 v[110:111], %[wb1] offset:41984\n\t" \
    "s_barrier\n\t" \
    "v_mfma_f32_16x16x128_f8f6f4 v[20:23], v[136:143], v[48:55], v[20:23]\n\t" \
    "ds_read_b128 v[48:51], %[ab0] offset:0\n\t" \
    "s_waitcnt lgkmcnt(10)\n\t" \
    "v_mfma_f32_16x16x128_f8f6f4 v[24:27], v[128:135], v[56:63], v[24:27]\n\t" \
    "ds_read_b128 v[52:55], %[ab1] offset:0\n\t" \
    "v_mfma_f32_16x16x128_f8f6f4 v[28:31], v[136:143], v[56:63], v[28:31]\n\t" \
    "ds_read_b128 v[56:59], %[ab0] offset:2048\n\t" \
    "s_waitcnt lgkmcnt(8)\n\t" \
    "v_mfma_f32_16x16x128_f8f6f4 v[32:35], v[128:135], v[64:71], v[32:35]\n\t" \
    "ds_read_b128 v[60:63], %[ab1] offset:2048\n\t" \
    "v_mfma_f32_16x16x128_f8f6f4 v[36:39], v[136:143], v[64:71], v[36:39]\n\t" \
    "ds_read_b128 v[64:67], %[ab0] offset:4096\n\t" \
    "s_waitcnt lgkmcnt(6)\n\t" \
    "v_mfma_f32_16x16x128_f8f6f4 v[40:43], v[128:135], v[72:79], v[40:43]\n\t" \
    "ds_read_b128 v[68:71], %[ab1] offset:4096\n\t" \
    "v_mfma_f32_16x16x128_f8f6f4 v[44:47], v[136:143], v[72:79], v[44:47]\n\t" \
    "ds_read_b128 v[72:75], %[ab0] offset:6144\n\t" \
    "s_waitcnt lgkmcnt(5)\n\t" \
    "v_mfma_f32_16x16x128_f8f6f4 v[16:19], v[144:151], v[48:55], v[16:19]\n\t" \
    "ds_read_b128 v[76:79], %[ab1] offset:6144\n\t" \
    "s_barrier\n\t" \
    "v_mfma_f32_16x16x128_f8f6f4 v[20:23], v[152:159], v[48:55], v[20:23]\n\t" \
    "ds_read_b128 v[48:51], %[ab0] offset:8192\n\t" \
    "ds_read_b64_tr_b8 v[112:113], %[wb0] offset:49152\n\t" \
    "s_waitcnt lgkmcnt(6)\n\t" \
    "v_mfma_f32_16x16x128_f8f6f4 v[24:27], v[144:151], v[56:63], v[24:27]\n\t" \
    "ds_read_b128 v[52:55], %[ab1] offset:8192\n\t" \
    "ds_read_b64_tr_b8 v[114:115], %[wb0] offset:50176\n\t" \
    "v_mfma_f32_16x16x128_f8f6f4 v[28:31], v[152:159], v[56:63], v[28:31]\n\t" \
    "ds_read_b128 v[56:59], %[ab0] offset:10240\n\t" \
    "ds_read_b64_tr_b8 v[116:117], %[wb0] offset:57344\n\t" \
    "s_waitcnt lgkmcnt(8)\n\t" \
    "v_mfma_f32_16x16x128_f8f6f4 v[32:35], v[144:151], v[64:71], v[32:35]\n\t" \
    "ds_read_b128 v[60:63], %[ab1] offset:10240\n\t" \
    "ds_read_b64_tr_b8 v[118:119], %[wb0] offset:58368\n\t" \
    "v_mfma_f32_16x16x128_f8f6f4 v[36:39], v[152:159], v[64:71], v[36:39]\n\t" \
    "ds_read_b128 v[64:67], %[ab0] offset:12288\n\t" \
    "ds_read_b64_tr_b8 v[120:121], %[wb1] offset:49152\n\t" \
    "s_waitcnt lgkmcnt(10)\n\t" \
    "v_mfma_f32_16x16x128_f8f6f4 v[40:43], v[144:151], v[72:79], v[40:43]\n\t" \
    "ds_read_b128 v[68:71], %[ab1] offset:12288\n\t" \
    "ds_read_b64_tr_b8 v[122:123], %[wb1] offset:50176\n\t" \
    "v_mfma_f32_16x16x128_f8f6f4 v[44:47], v[152:159], v[72:79], v[44:47]\n\t" \
    "ds_read_b128 v[72:75], %[ab0] offset:14336\n\t" \
    "ds_read_b64_tr_b8 v[124:125], %[wb1] offset:57344\n\t" \
    "s_waitcnt lgkmcnt(11)\n\t" \
    "v_mfma_f32_16x16x128_f8f6f4 v[16:19], v[160:167], v[48:55], v[16:19]\n\t" \
    "ds_read_b128 v[76:79], %[ab1] offset:14336\n\t" \
    "ds_read_b64_tr_b8 v[126:127], %[wb1] offset:58368\n\t" \
    "s_barrier\n\t" \
    "v_mfma_f32_16x16x128_f8f6f4 v[20:23], v[168:175], v[48:55], v[20:23]\n\t" \
    "ds_read_b128 v[48:51], %[ab0] offset:16384\n\t" \
    "s_waitcnt lgkmcnt(10)\n\t" \
    "v_mfma_f32_16x16x128_f8f6f4 v[24:27], v[160:167], v[56:63], v[24:27]\n\t" \
    "ds_read_b128 v[52:55], %[ab1] offset:16384\n\t" \
    "v_mfma_f32_16x16x128_f8f6f4 v[28:31], v[168:175], v[56:63], v[28:31]\n\t" \
    "ds_read_b128 v[56:59], %[ab0] offset:18432\n\t" \
    "s_waitcnt lgkmcnt(8)\n\t" \
    "v_mfma_f32_16x16x128_f8f6f4 v[32:35], v[160:167], v[64:71], v[32:35]\n\t" \
    "ds_read_b128 v[60:63], %[ab1] offset:18432\n\t" \
    "v_mfma_f32_16x16x128_f8f6f4 v[36:39], v[168:175], v[64:71], v[36:39]\n\t" \
    "ds_read_b128 v[64:67], %[ab0] offset:20480\n\t" \
    "s_waitcnt lgkmcnt(6)\n\t" \
    "v_mfma_f32_16x16x128_f8f6f4 v[40:43], v[160:167], v[72:79], v[40:43]\n\t" \
    "ds_read_b128 v[68:71], %[ab1] offset:20480\n\t" \
    "v_mfma_f32_16x16x128_f8f6f4 v[44:47], v[168:175], v[72:79], v[44:47]\n\t" \
    "ds_read_b128 v[72:75], %[ab0] offset:22528\n\t" \
    "s_waitcnt lgkmcnt(5)\n\t" \
    "v_mfma_f32_16x16x128_f8f6f4 v[16:19], v[176:183], v[48:55], v[16:19]\n\t" \
    "ds_read_b128 v[76:79], %[ab1] offset:22528\n\t" \
    "s_barrier\n\t" \
    "v_mfma_f32_16x16x128_f8f6f4 v[20:23], v[184:191], v[48:55], v[20:23]\n\t" \
    "ds_read_b128 v[48:51], %[ab0] offset:24576\n\t" \
    "ds_read_b64_tr_b8 v[128:129], %[wb0h] offset:0\n\t" \
    "s_waitcnt lgkmcnt(6)\n\t" \
    "v_mfma_f32_16x16x128_f8f6f4 v[24:27], v[176:183], v[56:63], v[24:27]\n\t" \
    "ds_read_b128 v[52:55], %[ab1] offset:24576\n\t" \
    "ds_read_b64_tr_b8 v[130:131], %[wb0h] offset:1024\n\t" \
    "v_mfma_f32_16x16x128_f8f6f4 v[28:31], v[184:191], v[56:63], v[28:31]\n\t" \
    "ds_read_b128 v[56:59], %[ab0] offset:26624\n\t" \
    "ds_read_b64_tr_b8 v[132:133], %[wb0h] offset:8192\n\t" \
    "s_waitcnt lgkmcnt(8)\n\t" \
    "v_mfma_f32_16x16x128_f8f6f4 v[32:35], v[176:183], v[64:71], v[32:35]\n\t" \
    "ds_read_b128 v[60:63], %[ab1] offset:26624\n\t" \
    "ds_read_b64_tr_b8 v[134:135], %[wb0h] offset:9216\n\t" \
    "v_mfma_f32_16x16x128_f8f6f4 v[36:39], v[184:191], v[64:71], v[36:39]\n\t" \
    "ds_read_b128 v[64:67], %[ab0] offset:28672\n\t" \
    "ds_read_b64_tr_b8 v[136:137], %[wb1h] offset:0\n\t" \
    "s_waitcnt lgkmcnt(10)\n\t" \
    "v_mfma_f32_16x16x128_f8f6f4 v[40:43], v[176:183], v[72:79], v[40:43]\n\t" \
    "ds_read_b128 v[68:71], %[ab1] offset:28672\n\t" \
    "ds_read_b64_tr_b8 v[138:139], %[wb1h] offset:1024\n\t" \
    "v_mfma_f32_16x16x128_f8f6f4 v[44:47], v[184:191], v[72:79], v[44:47]\n\t" \
    "ds_read_b128 v[72:75], %[ab0] offset:30720\n\t" \
    "ds_read_b64_tr_b8 v[140:141], %[wb1h] offset:8192\n\t" \
    "s_waitcnt lgkmcnt(11)\n\t" \
    "v_mfma_f32_16x16x128_f8f6f4 v[16:19], v[192:199], v[48:55], v[16:19]\n\t" \
    "ds_read_b128 v[76:79], %[ab1] offset:30720\n\t" \
    "ds_read_b64_tr_b8 v[142:143], %[wb1h] offset:9216\n\t" \
    "s_barrier\n\t" \
    "v_mfma_f32_16x16x128_f8f6f4 v[20:23], v[200:207], v[48:55], v[20:23]\n\t" \
    "ds_read_b128 v[48:51], %[ab0] offset:32768\n\t" \
    "s_waitcnt lgkmcnt(10)\n\t" \
    "v_mfma_f32_16x16x128_f8f6f4 v[24:27], v[192:199], v[56:63], v[24:27]\n\t" \
    "ds_read_b128 v[52:55], %[ab1] offset:32768\n\t" \
    "v_mfma_f32_16x16x128_f8f6f4 v[28:31], v[200:207], v[56:63], v[28:31]\n\t" \
    "ds_read_b128 v[56:59], %[ab0] offset:34816\n\t" \
    "s_waitcnt lgkmcnt(8)\n\t" \
    "v_mfma_f32_16x16x128_f8f6f4 v[32:35], v[192:199], v[64:71], v[32:35]\n\t" \
    "ds_read_b128 v[60:63], %[ab1] offset:34816\n\t" \
    "v_mfma_f32_16x16x128_f8f6f4 v[36:39], v[200:207], v[64:71], v[36:39]\n\t" \
    "ds_read_b128 v[64:67], %[ab0] offset:36864\n\t" \
    "s_waitcnt lgkmcnt(6)\n\t" \
    "v_mfma_f32_16x16x128_f8f6f4 v[40:43], v[192:199], v[72:79], v[40:43]\n\t" \
    "ds_read_b128 v[68:71], %[ab1] offset:36864\n\t" \
    "v_mfma_f32_16x16x128_f8f6f4 v[44:47], v[200:207], v[72:79], v[44:47]\n\t" \
    "ds_read_b128 v[72:75], %[ab0] offset:38912\n\t" \
    "s_waitcnt lgkmcnt(5)\n\t" \
    "v_mfma_f32_16x16x128_f8f6f4 v[16:19], v[208:215], v[48:55], v[16:19]\n\t" \
    "ds_read_b128 v[76:79], %[ab1] offset:38912\n\t" \
    "s_barrier\n\t" \
    "v_mfma_f32_16x16x128_f8f6f4 v[20:23], v[216:223], v[48:55], v[20:23]\n\t" \
    "ds_read_b128 v[48:51], %[ab0] offset:0\n\t" \
    "ds_read_b64_tr_b8 v[144:145], %[wb0h] offset:16384\n\t" \
    "s_waitcnt lgkmcnt(6)\n\t" \
    "v_mfma_f32_16x16x128_f8f6f4 v[24:27], v[208:215], v[56:63], v[24:27]\n\t" \
    "ds_read_b128 v[52:55], %[ab1] offset:0\n\t" \
    "ds_read_b64_tr_b8 v[146:147], %[wb0h] offset:17408\n\t" \
    "v_mfma_f32_16x16x128_f8f6f4 v[28:31], v[216:223], v[56:63], v[28:31]\n\t" \
    "ds_read_b128 v[56:59], %[ab0] offset:2048\n\t" \
    "ds_read_b64_tr_b8 v[148:149], %[wb0h] offset:24576\n\t" \
    "s_waitcnt lgkmcnt(8)\n\t" \
    "v_mfma_f32_16x16x128_f8f6f4 v[32:35], v[208:215], v[64:71], v[32:35]\n\t" \
    "ds_read_b128 v[60:63], %[ab1] offset:2048\n\t" \
    "ds_read_b64_tr_b8 v[150:151], %[wb0h] offset:25600\n\t" \
    "v_mfma_f32_16x16x128_f8f6f4 v[36:39], v[216:223], v[64:71], v[36:39]\n\t" \
    "ds_read_b128 v[64:67], %[ab0] offset:4096\n\t" \
    "ds_read_b64_tr_b8 v[152:153], %[wb1h] offset:16384\n\t" \
    "s_waitcnt lgkmcnt(10)\n\t" \
    "v_mfma_f32_16x16x128_f8f6f4 v[40:43], v[208:215], v[72:79], v[40:43]\n\t" \
    "ds_read_b128 v[68:71], %[ab1] offset:4096\n\t" \
    "ds_read_b64_tr_b8 v[154:155], %[wb1h] offset:17408\n\t" \
    "v_mfma_f32_16x16x128_f8f6f4 v[44:47], v[216:223], v[72:79], v[44:47]\n\t" \
    "ds_read_b128 v[72:75], %[ab0] offset:6144\n\t" \
    "ds_read_b64_tr_b8 v[156:157], %[wb1h] offset:24576\n\t" \
    "s_waitcnt lgkmcnt(11)\n\t" \
    "v_mfma_f32_16x16x128_f8f6f4 v[16:19], v[224:231], v[48:55], v[16:19]\n\t" \
    "ds_read_b128 v[76:79], %[ab1] offset:6144\n\t" \
    "ds_read_b64_tr_b8 v[158:159], %[wb1h] offset:25600\n\t" \
    "s_barrier\n\t" \
    "v_mfma_f32_16x16x128_f8f6f4 v[20:23], v[232:239], v[48:55], v[20:23]\n\t" \
    "ds_read_b128 v[48:51], %[ab0] offset:8192\n\t" \
    "s_waitcnt lgkmcnt(10)\n\t" \
    "v_mfma_f32_16x16x128_f8f6f4 v[24:27], v[224:231], v[56:63], v[24:27]\n\t" \
    "ds_read_b128 v[52:55], %[ab1] offset:8192\n\t" \
    "v_mfma_f32_16x16x128_f8f6f4 v[28:31], v[232:239], v[56:63], v[28:31]\n\t" \
    "ds_read_b128 v[56:59], %[ab0] offset:10240\n\t" \
    "s_waitcnt lgkmcnt(8)\n\t" \
    "v_mfma_f32_16x16x128_f8f6f4 v[32:35], v[224:231], v[64:71], v[32:35]\n\t" \
    "ds_read_b128 v[60:63], %[ab1] offset:10240\n\t" \
    "v_mfma_f32_16x16x128_f8f6f4 v[36:39], v[232:239], v[64:71], v[36:39]\n\t" \
    "ds_read_b128 v[64:67], %[ab0] offset:12288\n\t" \
    "s_waitcnt lgkmcnt(6)\n\t" \
    "v_mfma_f32_16x16x128_f8f6f4 v[40:43], v[224:231], v[72:79], v[40:43]\n\t" \
    "ds_read_b128 v[68:71], %[ab1] offset:12288\n\t" \
    "v_mfma_f32_16x16x128_f8f6f4 v[44:47], v[232:239], v[72:79], v[44:47]\n\t" \
    "ds_read_b128 v[72:75], %[ab0] offset:14336\n\t" \
    "s_waitcnt lgkmcnt(5)\n\t" \
    "v_mfma_f32_16x16x128_f8f6f4 v[16:19], v[240:247], v[48:55], v[16:19]\n\t" \
    "ds_read_b128 v[76:79], %[ab1] offset:14336\n\t" \
    "s_barrier\n\t" \
    "v_mfma_f32_16x16x128_f8f6f4 v[20:23], v[248:255], v[48:55], v[20:23]\n\t" \
    "ds_read_b128 v[48:51], %[ab0] offset:16384\n\t" \
    "ds_read_b64_tr_b8 v[160:161], %[wb0h] offset:32768\n\t" \
    "s_waitcnt lgkmcnt(6)\n\t" \
    "v_mfma_f32_16x16x128_f8f6f4 v[24:27], v[240:247], v[56:63], v[24:27]\n\t" \
    "ds_read_b128 v[52:55], %[ab1] offset:16384\n\t" \
    "ds_read_b64_tr_b8 v[162:163], %[wb0h] offset:33792\n\t" \
    "v_mfma_f32_16x16x128_f8f6f4 v[28:31], v[248:255], v[56:63], v[28:31]\n\t" \
    "ds_read_b128 v[56:59], %[ab0] offset:18432\n\t" \
    "ds_read_b64_tr_b8 v[164:165], %[wb0h] offset:40960\n\t" \
    "s_waitcnt lgkmcnt(8)\n\t" \
    "v_mfma_f32_16x16x128_f8f6f4 v[32:35], v[240:247], v[64:71], v[32:35]\n\t" \
    "ds_read_b128 v[60:63], %[ab1] offset:18432\n\t" \
    "ds_read_b64_tr_b8 v[166:167], %[wb0h] offset:41984\n\t" \
    "v_mfma_f32_16x16x128_f8f6f4 v[36:39], v[248:255], v[64:71], v[36:39]\n\t" \
    "ds_read_b128 v[64:67], %[ab0] offset:20480\n\t" \
    "ds_read_b64_tr_b8 v[168:169], %[wb1h] offset:32768\n\t" \
    "s_waitcnt lgkmcnt(10)\n\t" \
    "v_mfma_f32_16x16x128_f8f6f4 v[40:43], v[240:247], v[72:79], v[40:43]\n\t" \
    "ds_read_b128 v[68:71], %[ab1] offset:20480\n\t" \
    "ds_read_b64_tr_b8 v[170:171], %[wb1h] offset:33792\n\t" \
    "v_mfma_f32_16x16x128_f8f6f4 v[44:47], v[248:255], v[72:79], v[44:47]\n\t" \
    "ds_read_b128 v[72:75], %[ab0] offset:22528\n\t" \
    "ds_read_b64_tr_b8 v[172:173], %[wb1h] offset:40960\n\t" \
    "s_waitcnt lgkmcnt(11)\n\t" \
    "v_mfma_f32_16x16x128_f8f6f4 v[16:19], v[80:87], v[48:55], v[16:19]\n\t" \
    "ds_read_b128 v[76:79], %[ab1] offset:22528\n\t" \
    "ds_read_b64_tr_b8 v[174:175], %[wb1h] offset:41984\n\t" \
    "s_barrier\n\t" \
    "v_mfma_f32_16x16x128_f8f6f4 v[20:23], v[88:95], v[48:55], v[20:23]\n\t" \
    "ds_read_b128 v[48:51], %[ab0] offset:24576\n\t" \
    "s_waitcnt lgkmcnt(10)\n\t" \
    "v_mfma_f32_16x16x128_f8f6f4 v[24:27], v[80:87], v[56:63], v[24:27]\n\t" \
    "ds_read_b128 v[52:55], %[ab1] offset:24576\n\t" \
    "v_mfma_f32_16x16x128_f8f6f4 v[28:31], v[88:95], v[56:63], v[28:31]\n\t" \
    "ds_read_b128 v[56:59], %[ab0] offset:26624\n\t" \
    "s_waitcnt lgkmcnt(8)\n\t" \
    "v_mfma_f32_16x16x128_f8f6f4 v[32:35], v[80:87], v[64:71], v[32:35]\n\t" \
    "ds_read_b128 v[60:63], %[ab1] offset:26624\n\t" \
    "v_mfma_f32_16x16x128_f8f6f4 v[36:39], v[88:95], v[64:71], v[36:39]\n\t" \
    "ds_read_b128 v[64:67], %[ab0] offset:28672\n\t" \
    "s_waitcnt lgkmcnt(6)\n\t" \
    "v_mfma_f32_16x16x128_f8f6f4 v[40:43], v[80:87], v[72:79], v[40:43]\n\t" \
    "ds_read_b128 v[68:71], %[ab1] offset:28672\n\t" \
    "v_mfma_f32_16x16x128_f8f6f4 v[44:47], v[88:95], v[72:79], v[44:47]\n\t" \
    "ds_read_b128 v[72:75], %[ab0] offset:30720\n\t" \
    "s_waitcnt lgkmcnt(5)\n\t" \
    "v_mfma_f32_16x16x128_f8f6f4 v[16:19], v[96:103], v[48:55], v[16:19]\n\t" \
    "ds_read_b128 v[76:79], %[ab1] offset:30720\n\t" \
    "s_barrier\n\t" \
    "v_mfma_f32_16x16x128_f8f6f4 v[20:23], v[104:111], v[48:55], v[20:23]\n\t" \
    "ds_read_b128 v[48:51], %[ab0] offset:32768\n\t" \
    "ds_read_b64_tr_b8 v[176:177], %[wb0] offset:0\n\t" \
    "s_waitcnt lgkmcnt(6)\n\t" \
    "v_mfma_f32_16x16x128_f8f6f4 v[24:27], v[96:103], v[56:63], v[24:27]\n\t" \
    "ds_read_b128 v[52:55], %[ab1] offset:32768\n\t" \
    "ds_read_b64_tr_b8 v[178:179], %[wb0] offset:1024\n\t" \
    "v_mfma_f32_16x16x128_f8f6f4 v[28:31], v[104:111], v[56:63], v[28:31]\n\t" \
    "ds_read_b128 v[56:59], %[ab0] offset:34816\n\t" \
    "ds_read_b64_tr_b8 v[180:181], %[wb0] offset:8192\n\t" \
    "s_waitcnt lgkmcnt(8)\n\t" \
    "v_mfma_f32_16x16x128_f8f6f4 v[32:35], v[96:103], v[64:71], v[32:35]\n\t" \
    "ds_read_b128 v[60:63], %[ab1] offset:34816\n\t" \
    "ds_read_b64_tr_b8 v[182:183], %[wb0] offset:9216\n\t" \
    "v_mfma_f32_16x16x128_f8f6f4 v[36:39], v[104:111], v[64:71], v[36:39]\n\t" \
    "ds_read_b128 v[64:67], %[ab0] offset:36864\n\t" \
    "ds_read_b64_tr_b8 v[184:185], %[wb1] offset:0\n\t" \
    "s_waitcnt lgkmcnt(10)\n\t" \
    "v_mfma_f32_16x16x128_f8f6f4 v[40:43], v[96:103], v[72:79], v[40:43]\n\t" \
    "ds_read_b128 v[68:71], %[ab1] offset:36864\n\t" \
    "ds_read_b64_tr_b8 v[186:187], %[wb1] offset:1024\n\t" \
    "v_mfma_f32_16x16x128_f8f6f4 v[44:47], v[104:111], v[72:79], v[44:47]\n\t" \
    "ds_read_b128 v[72:75], %[ab0] offset:38912\n\t" \
    "ds_read_b64_tr_b8 v[188:189], %[wb1] offset:8192\n\t" \
    "s_waitcnt lgkmcnt(11)\n\t" \
    "v_mfma_f32_16x16x128_f8f6f4 v[16:19], v[112:119], v[48:55], v[16:19]\n\t" \
    "ds_read_b128 v[76:79], %[ab1] offset:38912\n\t" \
    "ds_read_b64_tr_b8 v[190:191], %[wb1] offset:9216\n\t" \
    "s_barrier\n\t" \
    "v_mfma_f32_16x16x128_f8f6f4 v[20:23], v[120:127], v[48:55], v[20:23]\n\t" \
    "ds_read_b128 v[48:51], %[ab0] offset:0\n\t" \
    "s_waitcnt lgkmcnt(10)\n\t" \
    "v_mfma_f32_16x16x128_f8f6f4 v[24:27], v[112:119], v[56:63], v[24:27]\n\t" \
    "ds_read_b128 v[52:55], %[ab1] offset:0\n\t" \
    "v_mfma_f32_16x16x128_f8f6f4 v[28:31], v[120:127], v[56:63], v[28:31]\n\t" \
    "ds_read_b128 v[56:59], %[ab0] offset:2048\n\t" \
    "s_waitcnt lgkmcnt(8)\n\t" \
    "v_mfma_f32_16x16x128_f8f6f4 v[32:35], v[112:119], v[64:71], v[32:35]\n\t" \
    "ds_read_b128 v[60:63], %[ab1] offset:2048\n\t" \
    "v_mfma_f32_16x16x128_f8f6f4 v[36:39], v[120:127], v[64:71], v[36:39]\n\t" \
    "ds_read_b128 v[64:67], %[ab0] offset:4096\n\t" \
    "s_waitcnt lgkmcnt(6)\n\t" \
    "v_mfma_f32_16x16x128_f8f6f4 v[40:43], v[112:119], v[72:79], v[40:43]\n\t" \
    "ds_read_b128 v[68:71], %[ab1] offset:4096\n\t" \
    "v_mfma_f32_16x16x128_f8f6f4 v[44:47], v[120:127], v[72:79], v[44:47]\n\t" \
    "ds_read_b128 v[72:75], %[ab0] offset:6144\n\t" \
    "s_waitcnt lgkmcnt(5)\n\t" \
    "v_mfma_f32_16x16x128_f8f6f4 v[16:19], v[128:135], v[48:55], v[16:19]\n\t" \
    "ds_read_b128 v[76:79], %[ab1] offset:6144\n\t" \
    "s_barrier\n\t" \
    "v_mfma_f32_16x16x128_f8f6f4 v[20:23], v[136:143], v[48:55], v[20:23]\n\t" \
    "ds_read_b128 v[48:51], %[ab0] offset:8192\n\t" \
    "ds_read_b64_tr_b8 v[192:193], %[wb0] offset:16384\n\t" \
    "s_waitcnt lgkmcnt(6)\n\t" \
    "v_mfma_f32_16x16x128_f8f6f4 v[24:27], v[128:135], v[56:63], v[24:27]\n\t" \
    "ds_read_b128 v[52:55], %[ab1] offset:8192\n\t" \
    "ds_read_b64_tr_b8 v[194:195], %[wb0] offset:17408\n\t" \
    "v_mfma_f32_16x16x128_f8f6f4 v[28:31], v[136:143], v[56:63], v[28:31]\n\t" \
    "ds_read_b128 v[56:59], %[ab0] offset:10240\n\t" \
    "ds_read_b64_tr_b8 v[196:197], %[wb0] offset:24576\n\t" \
    "s_waitcnt lgkmcnt(8)\n\t" \
    "v_mfma_f32_16x16x128_f8f6f4 v[32:35], v[128:135], v[64:71], v[32:35]\n\t" \
    "ds_read_b128 v[60:63], %[ab1] offset:10240\n\t" \
    "ds_read_b64_tr_b8 v[198:199], %[wb0] offset:25600\n\t" \
    "v_mfma_f32_16x16x128_f8f6f4 v[36:39], v[136:143], v[64:71], v[36:39]\n\t" \
    "ds_read_b128 v[64:67], %[ab0] offset:12288\n\t" \
    "ds_read_b64_tr_b8 v[200:201], %[wb1] offset:16384\n\t" \
    "s_waitcnt lgkmcnt(10)\n\t" \
    "v_mfma_f32_16x16x128_f8f6f4 v[40:43], v[128:135], v[72:79], v[40:43]\n\t" \
    "ds_read_b128 v[68:71], %[ab1] offset:12288\n\t" \
    "ds_read_b64_tr_b8 v[202:203], %[wb1] offset:17408\n\t" \
    "v_mfma_f32_16x16x128_f8f6f4 v[44:47], v[136:143], v[72:79], v[44:47]\n\t" \
    "ds_read_b128 v[72:75], %[ab0] offset:14336\n\t" \
    "ds_read_b64_tr_b8 v[204:205], %[wb1] offset:24576\n\t" \
    "s_waitcnt lgkmcnt(11)\n\t" \
    "v_mfma_f32_16x16x128_f8f6f4 v[16:19], v[144:151], v[48:55], v[16:19]\n\t" \
    "ds_read_b128 v[76:79], %[ab1] offset:14336\n\t" \
    "ds_read_b64_tr_b8 v[206:207], %[wb1] offset:25600\n\t" \
    "s_barrier\n\t" \
    "v_mfma_f32_16x16x128_f8f6f4 v[20:23], v[152:159], v[48:55], v[20:23]\n\t" \
    "ds_read_b128 v[48:51], %[ab0] offset:16384\n\t" \
    "s_waitcnt lgkmcnt(10)\n\t" \
    "v_mfma_f32_16x16x128_f8f6f4 v[24:27], v[144:151], v[56:63], v[24:27]\n\t" \
    "ds_read_b128 v[52:55], %[ab1] offset:16384\n\t" \
    "v_mfma_f32_16x16x128_f8f6f4 v[28:31], v[152:159], v[56:63], v[28:31]\n\t" \
    "ds_read_b128 v[56:59], %[ab0] offset:18432\n\t" \
    "s_waitcnt lgkmcnt(8)\n\t" \
    "v_mfma_f32_16x16x128_f8f6f4 v[32:35], v[144:151], v[64:71], v[32:35]\n\t" \
    "ds_read_b128 v[60:63], %[ab1] offset:18432\n\t" \
    "v_mfma_f32_16x16x128_f8f6f4 v[36:39], v[152:159], v[64:71], v[36:39]\n\t" \
    "ds_read_b128 v[64:67], %[ab0] offset:20480\n\t" \
    "s_waitcnt lgkmcnt(6)\n\t" \
    "v_mfma_f32_16x16x128_f8f6f4 v[40:43], v[144:151], v[72:79], v[40:43]\n\t" \
    "ds_read_b128 v[68:71], %[ab1] offset:20480\n\t" \
    "v_mfma_f32_16x16x128_f8f6f4 v[44:47], v[152:159], v[72:79], v[44:47]\n\t" \
    "ds_read_b128 v[72:75], %[ab0] offset:22528\n\t" \
    "s_waitcnt lgkmcnt(5)\n\t" \
    "v_mfma_f32_16x16x128_f8f6f4 v[16:19], v[160:167], v[48:55], v[16:19]\n\t" \
    "ds_read_b128 v[76:79], %[ab1] offset:22528\n\t" \
    "s_barrier\n\t" \
    "v_mfma_f32_16x16x128_f8f6f4 v[20:23], v[168:175], v[48:55], v[20:23]\n\t" \
    "ds_read_b128 v[48:51], %[ab0] offset:24576\n\t" \
    "ds_read_b64_tr_b8 v[208:209], %[wb0] offset:32768\n\t" \
    "s_waitcnt lgkmcnt(6)\n\t" \
    "v_mfma_f32_16x16x128_f8f6f4 v[24:27], v[160:167], v[56:63], v[24:27]\n\t" \
    "ds_read_b128 v[52:55], %[ab1] offset:24576\n\t" \
    "ds_read_b64_tr_b8 v[210:211], %[wb0] offset:33792\n\t" \
    "v_mfma_f32_16x16x128_f8f6f4 v[28:31], v[168:175], v[56:63], v[28:31]\n\t" \
    "ds_read_b128 v[56:59], %[ab0] offset:26624\n\t" \
    "ds_read_b64_tr_b8 v[212:213], %[wb0] offset:40960\n\t" \
    "s_waitcnt lgkmcnt(8)\n\t" \
    "v_mfma_f32_16x16x128_f8f6f4 v[32:35], v[160:167], v[64:71], v[32:35]\n\t" \
    "ds_read_b128 v[60:63], %[ab1] offset:26624\n\t" \
    "ds_read_b64_tr_b8 v[214:215], %[wb0] offset:41984\n\t" \
    "v_mfma_f32_16x16x128_f8f6f4 v[36:39], v[168:175], v[64:71], v[36:39]\n\t" \
    "ds_read_b128 v[64:67], %[ab0] offset:28672\n\t" \
    "ds_read_b64_tr_b8 v[216:217], %[wb1] offset:32768\n\t" \
    "s_waitcnt lgkmcnt(10)\n\t" \
    "v_mfma_f32_16x16x128_f8f6f4 v[40:43], v[160:167], v[72:79], v[40:43]\n\t" \
    "ds_read_b128 v[68:71], %[ab1] offset:28672\n\t" \
    "ds_read_b64_tr_b8 v[218:219], %[wb1] offset:33792\n\t" \
    "v_mfma_f32_16x16x128_f8f6f4 v[44:47], v[168:175], v[72:79], v[44:47]\n\t" \
    "ds_read_b128 v[72:75], %[ab0] offset:30720\n\t" \
    "ds_read_b64_tr_b8 v[220:221], %[wb1] offset:40960\n\t" \
    "s_waitcnt lgkmcnt(11)\n\t" \
    "v_mfma_f32_16x16x128_f8f6f4 v[16:19], v[176:183], v[48:55], v[16:19]\n\t" \
    "ds_read_b128 v[76:79], %[ab1] offset:30720\n\t" \
    "ds_read_b64_tr_b8 v[222:223], %[wb1] offset:41984\n\t" \
    "s_barrier\n\t" \
    "v_mfma_f32_16x16x128_f8f6f4 v[20:23], v[184:191], v[48:55], v[20:23]\n\t" \
    "ds_read_b128 v[48:51], %[ab0] offset:32768\n\t" \
    "s_waitcnt lgkmcnt(10)\n\t" \
    "v_mfma_f32_16x16x128_f8f6f4 v[24:27], v[176:183], v[56:63], v[24:27]\n\t" \
    "ds_read_b128 v[52:55], %[ab1] offset:32768\n\t" \
    "v_mfma_f32_16x16x128_f8f6f4 v[28:31], v[184:191], v[56:63], v[28:31]\n\t" \
    "ds_read_b128 v[56:59], %[ab0] offset:34816\n\t" \
    "s_waitcnt lgkmcnt(8)\n\t" \
    "v_mfma_f32_16x16x128_f8f6f4 v[32:35], v[176:183], v[64:71], v[32:35]\n\t" \
    "ds_read_b128 v[60:63], %[ab1] offset:34816\n\t" \
    "v_mfma_f32_16x16x128_f8f6f4 v[36:39], v[184:191], v[64:71], v[36:39]\n\t" \
    "ds_read_b128 v[64:67], %[ab0] offset:36864\n\t" \
    "s_waitcnt lgkmcnt(6)\n\t" \
    "v_mfma_f32_16x16x128_f8f6f4 v[40:43], v[176:183], v[72:79], v[40:43]\n\t" \
    "ds_read_b128 v[68:71], %[ab1] offset:36864\n\t" \
    "v_mfma_f32_16x16x128_f8f6f4 v[44:47], v[184:191], v[72:79], v[44:47]\n\t" \
    "ds_read_b128 v[72:75], %[ab0] offset:38912\n\t" \
    "s_waitcnt lgkmcnt(5)\n\t" \
    "v_mfma_f32_16x16x128_f8f6f4 v[16:19], v[192:199], v[48:55], v[16:19]\n\t" \
    "ds_read_b128 v[76:79], %[ab1] offset:38912\n\t" \
    "s_barrier\n\t" \
    "v_mfma_f32_16x16x128_f8f6f4 v[20:23], v[200:207], v[48:55], v[20:23]\n\t" \
    "ds_read_b128 v[48:51], %[ab0] offset:0\n\t" \
    "ds_read_b64_tr_b8 v[224:225], %[wb0] offset:49152\n\t" \
    "s_waitcnt lgkmcnt(6)\n\t" \
    "v_mfma_f32_16x16x128_f8f6f4 v[24:27], v[192:199], v[56:63], v[24:27]\n\t" \
    "ds_read_b128 v[52:55], %[ab1] offset:0\n\t" \
    "ds_read_b64_tr_b8 v[226:227], %[wb0] offset:50176\n\t" \
    "v_mfma_f32_16x16x128_f8f6f4 v[28:31], v[200:207], v[56:63], v[28:31]\n\t" \
    "ds_read_b128 v[56:59], %[ab0] offset:2048\n\t" \
    "ds_read_b64_tr_b8 v[228:229], %[wb0] offset:57344\n\t" \
    "s_waitcnt lgkmcnt(8)\n\t" \
    "v_mfma_f32_16x16x128_f8f6f4 v[32:35], v[192:199], v[64:71], v[32:35]\n\t" \
    "ds_read_b128 v[60:63], %[ab1] offset:2048\n\t" \
    "ds_read_b64_tr_b8 v[230:231], %[wb0] offset:58368\n\t" \
    "v_mfma_f32_16x16x128_f8f6f4 v[36:39], v[200:207], v[64:71], v[36:39]\n\t" \
    "ds_read_b128 v[64:67], %[ab0] offset:4096\n\t" \
    "ds_read_b64_tr_b8 v[232:233], %[wb1] offset:49152\n\t" \
    "s_waitcnt lgkmcnt(10)\n\t" \
    "v_mfma_f32_16x16x128_f8f6f4 v[40:43], v[192:199], v[72:79], v[40:43]\n\t" \
    "ds_read_b128 v[68:71], %[ab1] offset:4096\n\t" \
    "ds_read_b64_tr_b8 v[234:235], %[wb1] offset:50176\n\t" \
    "v_mfma_f32_16x16x128_f8f6f4 v[44:47], v[200:207], v[72:79], v[44:47]\n\t" \
    "ds_read_b128 v[72:75], %[ab0] offset:6144\n\t" \
    "ds_read_b64_tr_b8 v[236:237], %[wb1] offset:57344\n\t" \
    "s_waitcnt lgkmcnt(11)\n\t" \
    "v_mfma_f32_16x16x128_f8f6f4 v[16:19], v[208:215], v[48:55], v[16:19]\n\t" \
    "ds_read_b128 v[76:79], %[ab1] offset:6144\n\t" \
    "ds_read_b64_tr_b8 v[238:239], %[wb1] offset:58368\n\t" \
    "v_mfma_f32_16x16x128_f8f6f4 v[20:23], v[216:223], v[48:55], v[20:23]\n\t" \
    "s_waitcnt lgkmcnt(9)\n\t" \
    "v_mfma_f32_16x16x128_f8f6f4 v[24:27], v[208:215], v[56:63], v[24:27]\n\t" \
    "v_mfma_f32_16x16x128_f8f6f4 v[28:31], v[216:223], v[56:63], v[28:31]\n\t" \
    "s_waitcnt lgkmcnt(5)\n\t" \
    "v_mfma_f32_16x16x128_f8f6f4 v[32:35], v[208:215], v[64:71], v[32:35]\n\t" \
    "v_mfma_f32_16x16x128_f8f6f4 v[36:39], v[216:223], v[64:71], v[36:39]\n\t" \
    "s_waitcnt lgkmcnt(1)\n\t" \
    "v_mfma_f32_16x16x128_f8f6f4 v[40:43], v[208:215], v[72:79], v[40:43]\n\t" \
    "v_mfma_f32_16x16x128_f8f6f4 v[44:47], v[216:223], v[72:79], v[44:47]\n\t" \
    "s_barrier\n\t" \
    "v_mov_b32 v82, %[ctl]\n\t" \
    "ds_read_b64 v[80:81], v82\n\t" \
    "v_mov_b32 v83, %[sx]\n\t" \
    "v_mul_f32 v83, %[sw], v83\n\t" \
    "ds_read_b128 v[48:51], %[ab0] offset:8192\n\t" \
    "ds_read_b128 v[52:55], %[ab1] offset:8192\n\t" \
    "ds_read_b128 v[56:59], %[ab0] offset:10240\n\t" \
    "ds_read_b128 v[60:63], %[ab1] offset:10240\n\t" \
    "ds_read_b128 v[64:67], %[ab0] offset:12288\n\t" \
    "ds_read_b128 v[68:71], %[ab1] offset:12288\n\t" \
    "ds_read_b128 v[72:75], %[ab0] offset:14336\n\t" \
    "ds_read_b128 v[76:79], %[ab1] offset:14336\n\t" \
    "s_waitcnt lgkmcnt(8)\n\t" \
    "v_cmp_eq_u32 vcc, 0, v80\n\t" \
    "v_mov_b32 v80, 0x7fc00000\n\t" \
    "s_nop 1\n\t" \
    "v_cndmask_b32 v83, v80, v83, vcc\n\t" \
    "s_waitcnt lgkmcnt(6)\n\t" \
    "v_mfma_f32_16x16x128_f8f6f4 v[16:19], v[224:231], v[48:55], v[16:19]\n\t" \
    "v_mfma_f32_16x16x128_f8f6f4 v[20:23], v[232:239], v[48:55], v[20:23]\n\t" \
    "s_waitcnt lgkmcnt(4)\n\t" \
    "v_mfma_f32_16x16x128_f8f6f4 v[24:27], v[224:231], v[56:63], v[24:27]\n\t" \
    "v_mfma_f32_16x16x128_f8f6f4 v[28:31], v[232:239], v[56:63], v[28:31]\n\t" \
    "s_nop 7\n\t" \
    "v_mul_f32 v16, v83, v16\n\t" \
    "v_mul_f32 v17, v83, v17\n\t" \
    "v_mul_f32 v18, v83, v18\n\t" \
    "v_mul_f32 v19, v83, v19\n\t" \
    "global_store_dwordx4 %[oo], v[16:19], %[ob0] offset:0 sc1\n\t" \
    "s_waitcnt lgkmcnt(2)\n\t" \
    "v_mfma_f32_16x16x128_f8f6f4 v[32:35], v[224:231], v[64:71], v[32:35]\n\t" \
    "v_mul_f32 v20, v83, v20\n\t" \
    "v_mul_f32 v21, v83, v21\n\t" \
    "v_mul_f32 v22, v83, v22\n\t" \
    "v_mul_f32 v23, v83, v23\n\t" \
    "global_store_dwordx4 %[oo], v[20:23], %[ob0] offset:64 sc1\n\t" \
    "v_mfma_f32_16x16x128_f8f6f4 v[36:39], v[232:239], v[64:71], v[36:39]\n\t" \
    "v_mul_f32 v24, v83, v24\n\t" \
    "v_mul_f32 v25, v83, v25\n\t" \
    "v_mul_f32 v26, v83, v26\n\t" \
    "v_mul_f32 v27, v83, v27\n\t" \
    "global_store_dwordx4 %[oo], v[24:27], %[ob1] offset:0 sc1\n\t" \
    "s_waitcnt lgkmcnt(0)\n\t" \
    "v_mfma_f32_16x16x128_f8f6f4 v[40:43], v[224:231], v[72:79], v[40:43]\n\t" \
    "v_mul_f32 v28, v83, v28\n\t" \
    "v_mul_f32 v29, v83, v29\n\t" \
    "v_mul_f32 v30, v83, v30\n\t" \
    "v_mul_f32 v31, v83, v31\n\t" \
    "global_store_dwordx4 %[oo], v[28:31], %[ob1] offset:64 sc1\n\t" \
    "v_mfma_f32_16x16x128_f8f6f4 v[44:47], v[232:239], v[72:79], v[44:47]\n\t" \
    "v_mul_f32 v32, v83, v32\n\t" \
    "v_mul_f32 v33, v83, v33\n\t" \
    "v_mul_f32 v34, v83, v34\n\t" \
    "v_mul_f32 v35, v83, v35\n\t" \
    "global_store_dwordx4 %[oo], v[32:35], %[ob2] offset:0 sc1\n\t" \
    "v_mul_f32 v36, v83, v36\n\t" \
    "v_mul_f32 v37, v83, v37\n\t" \
    "v_mul_f32 v38, v83, v38\n\t" \
    "v_mul_f32 v39, v83, v39\n\t" \
    "global_store_dwordx4 %[oo], v[36:39], %[ob2] offset:64 sc1\n\t" \
    "v_mul_f32 v40, v83, v40\n\t" \
    "v_mul_f32 v41, v83, v41\n\t" \
    "v_mul_f32 v42, v83, v42\n\t" \
    "v_mul_f32 v43, v83, v43\n\t" \
    "global_store_dwordx4 %[oo], v[40:43], %[ob3] offset:0 sc1\n\t" \
    "v_mul_f32 v44, v83, v44\n\t" \
    "v_mul_f32 v45, v83, v45\n\t" \
    "v_mul_f32 v46, v83, v46\n\t" \
    "v_mul_f32 v47, v83, v47\n\t" \
    "global_store_dwordx4 %[oo], v[44:47], %[ob3] offset:64 sc1\n\t" \
    "v_mov_b32 v80, 0\n\t" \
    "global_store_dword v80, v81, %[cntp] sc0 sc1\n\t"

namespace {

constexpr int NR = 8;
constexpr int MS = 512;
constexpr int KC = 512;
constexpr int NN = 4096;
constexpr int KK = 4096;
constexpr int NWG = 256;
constexpr int NTHREADS = 512;
constexpr int NSTEP = 32;

constexpr int PACE2 = 29;
constexpr int HORIZON = 3;
constexpr int W_SLOT = 16384, A_SLOT = 8192;
constexpr int L_A = NW * W_SLOT;
constexpr int L_CTRL = L_A + NA * A_SLOT;
constexpr int LDS_BYTES = L_CTRL + 64;
static_assert(LDS_BYTES <= 160 * 1024, "LDS");

constexpr size_t WS_FLAG = 0;
constexpr size_t WS_CNT = 16384;
constexpr size_t WS_A = 1u << 20;
constexpr size_t A_BYTES = (size_t)MS * KK;
constexpr size_t WS_NEED = WS_A + 2 * A_BYTES;
static_assert(A_BYTES == 1u << 21 && KK * NN == 1 << 24, "shifts and masks in the gate");

constexpr unsigned DEADLINE_TICKS = 50000000u;

struct Params {
    unsigned char* ws[NR];
    const unsigned char* x;
    const unsigned char* w;
    unsigned char* my;
    float* out;
    const float* sx;
    const float* sw;
    int rank;
    int pad;
};

#define CLOBBER_16_255 \
    "v16", "v17", "v18", "v19", "v20", "v21", "v22", "v23", "v24", "v25", "v26", "v27", \
    "v28", "v29", "v30", "v31", "v32", "v33", "v34", "v35", "v36", "v37", "v38", "v39", \
    "v40", "v41", "v42", "v43", "v44", "v45", "v46", "v47", "v48", "v49", "v50", "v51", \
    "v52", "v53", "v54", "v55", "v56", "v57", "v58", "v59", "v60", "v61", "v62", "v63", \
    "v64", "v65", "v66", "v67", "v68", "v69", "v70", "v71", "v72", "v73", "v74", "v75", \
    "v76", "v77", "v78", "v79", "v80", "v81", "v82", "v83", "v84", "v85", "v86", "v87", \
    "v88", "v89", "v90", "v91", "v92", "v93", "v94", "v95", "v96", "v97", "v98", "v99", \
    "v100", "v101", "v102", "v103", "v104", "v105", "v106", "v107", "v108", "v109", "v110", "v111", \
    "v112", "v113", "v114", "v115", "v116", "v117", "v118", "v119", "v120", "v121", "v122", "v123", \
    "v124", "v125", "v126", "v127", "v128", "v129", "v130", "v131", "v132", "v133", "v134", "v135", \
    "v136", "v137", "v138", "v139", "v140", "v141", "v142", "v143", "v144", "v145", "v146", "v147", \
    "v148", "v149", "v150", "v151", "v152", "v153", "v154", "v155", "v156", "v157", "v158", "v159", \
    "v160", "v161", "v162", "v163", "v164", "v165", "v166", "v167", "v168", "v169", "v170", "v171", \
    "v172", "v173", "v174", "v175", "v176", "v177", "v178", "v179", "v180", "v181", "v182", "v183", \
    "v184", "v185", "v186", "v187", "v188", "v189", "v190", "v191", "v192", "v193", "v194", "v195", \
    "v196", "v197", "v198", "v199", "v200", "v201", "v202", "v203", "v204", "v205", "v206", "v207", \
    "v208", "v209", "v210", "v211", "v212", "v213", "v214", "v215", "v216", "v217", "v218", "v219", \
    "v220", "v221", "v222", "v223", "v224", "v225", "v226", "v227", "v228", "v229", "v230", "v231", \
    "v232", "v233", "v234", "v235", "v236", "v237", "v238", "v239", "v240", "v241", "v242", "v243", \
    "v244", "v245", "v246", "v247", "v248", "v249", "v250", "v251", "v252", "v253", "v254", "v255"
#define CLOBBER_128_223 \
    "v128", "v129", "v130", "v131", "v132", "v133", "v134", "v135", "v136", "v137", "v138", "v139", \
    "v140", "v141", "v142", "v143", "v144", "v145", "v146", "v147", "v148", "v149", "v150", "v151", \
    "v152", "v153", "v154", "v155", "v156", "v157", "v158", "v159", "v160", "v161", "v162", "v163", \
    "v164", "v165", "v166", "v167", "v168", "v169", "v170", "v171", "v172", "v173", "v174", "v175", \
    "v176", "v177", "v178", "v179", "v180", "v181", "v182", "v183", "v184", "v185", "v186", "v187", \
    "v188", "v189", "v190", "v191", "v192", "v193", "v194", "v195", "v196", "v197", "v198", "v199", \
    "v200", "v201", "v202", "v203", "v204", "v205", "v206", "v207", "v208", "v209", "v210", "v211", \
    "v212", "v213", "v214", "v215", "v216", "v217", "v218", "v219", "v220", "v221", "v222", "v223"

#define BAR() asm volatile("s_barrier" ::: "memory")
#define WVX(n) asm volatile("s_waitcnt vmcnt(%0)" :: "n"(n) : "memory")
#define DMA4(rs, lds, v0, v1, v2, v3, soff, aux) do { \
    __builtin_amdgcn_raw_ptr_buffer_load_lds(rs, (LAS void*)(lds), 16, v0, soff, 0, aux); \
    __builtin_amdgcn_raw_ptr_buffer_load_lds(rs, (LAS void*)(lds), 16, v1, soff, 1024, aux); \
    __builtin_amdgcn_raw_ptr_buffer_load_lds(rs, (LAS void*)(lds), 16, v2, soff, 2048, aux); \
    __builtin_amdgcn_raw_ptr_buffer_load_lds(rs, (LAS void*)(lds), 16, v3, soff, 3072, aux); } while (0)

__device__ __forceinline__ int step_block(int i, int m, int n) {
    const int j = i - 4, g = (j * 37) >> 8, jj = j - g * 7;
    const int pos = jj + (jj >= ((m + 7) & 7) ? 1 : 0);
    return i < 4 ? ((i + m + n) & 3) : (1 + ((pos - m) & 7)) * 4 + g;
}

}

__global__ void __launch_bounds__(NTHREADS, 2) a2a_gemm_kernel(Params p) {
    extern __shared__ __attribute__((aligned(16))) unsigned char smem[];
    LAS unsigned char* L = (LAS unsigned char*)smem;
    const unsigned long long tstart = __builtin_amdgcn_s_memrealtime();
    const int tid = threadIdx.x;
    const int wave = __builtin_amdgcn_readfirstlane(tid >> 6);
    const int lane = tid & 63;
    const int b = blockIdx.x;
    const int rank = p.rank;
    const unsigned char* const px = p.x;
    const unsigned char* const pwm = p.w;
    unsigned char* const my = p.my;
    unsigned char* const peer = p.ws[b & 7];
    float* const pout = p.out;
    const float* const psx = p.sx;
    const float* const psw = p.sw;
    const int mt = (b >> 3) & 7, nx = b >> 6;
    const int n0 = ((b & 7) * 4 + nx) * 128;
    int swz = ((lane & 7) ^ (lane >> 4)) * 16;
    int xoff = (lane >> 3) * KC + swz;
    int rel = step_block((wave == 0 ? lane + 6 + XD - T0 : lane) & 31, mt, nx);
    asm volatile("" : "+v"(swz), "+v"(xoff), "+v"(rel));
    asm volatile("" :: "s"(rank), "s"(px), "s"(pwm), "s"(my), "s"(peer), "s"(pout), "s"(psx), "s"(psw), "s"(tstart));
    const int tab = (rel + rank * 4) & 31;

    if (__builtin_expect(wave == 0, 1)) {
        __builtin_amdgcn_s_setprio(3);
        const int dst = b & 7;
        const int slot = mt * 4 + nx;
        const int q = (rank - dst - 1) & 7;
        const int push = dst ^ rank;
        const int multi = slot < 4 && push, serve = slot >= 4 && slot < 12;
        auto pg = [&](int z) { return z >> 3; };
        auto pm = [&](int z) { return (z - q) & 7; };
        auto xsrc = [&](int z) { return px + (size_t)((dst * MS + pm(z) * 64) * KC + pg(z) * 128); };
        auto qoff = [&](int z) { return (int)WS_A + (pm(z) * 32 + rank * 4 + pg(z)) * A_SLOT + 4096 + lane * 16; };
        auto fpeer = [&](int z) { return (int)WS_FLAG + pm(z) * 128 + (rank * 4 + pg(z)) * 4; };
#define XL2(r, a, c) "global_load_dwordx4 v[" #r ":" #r "+3], " a ", %8\n\tglobal_load_dwordx4 v[" #r "+4:" #r "+7], " c ", %8\n\t"
#define XLOAD(r, z) asm volatile(XL2(r, "%0", "%1") XL2(r+8, "%2", "%3") XL2(r+16, "%4", "%5") XL2(r+24, "%6", "%7") \
            :: "v"(xoff), "v"(x1), "v"(xoff + 0x2000), "v"(x1 + 0x2000), "v"(xoff + 0x4000), "v"(x1 + 0x4000), \
               "v"(xoff + 0x6000), "v"(x1 + 0x6000), "s"(xsrc(z)) : "memory", CLOBBER_128_223)
        const int x1 = (xoff ^ 64) + 8 * KC;
        if (!serve) XLOAD(128, slot);
        asm volatile("global_load_dword v252, %0, %1 sc0 sc1"
                     :: "v"(0), "s"(my + WS_CNT + (size_t)b * 128) : "memory", "v252");
        if (!multi && !serve) BAR();
        const int tmo = (int)WS_FLAG + mt * 128 + rank * 16;
        const int foff = ((KNOWN | 1ull << 63) >> lane & 1) ? tmo : (int)WS_FLAG + mt * 128 + tab * 4;
        asm volatile("global_load_dword v250, %0, %1 sc0 sc1" :: "v"(foff), "s"(my) : "memory", "v250");
        const int pr0 = serve ? (slot - 4) * 16 : slot < 4 ? KC : 256 + (slot - 12) * 16;
        int pf = pr0 < KC ? ((rank + 1) * KC + pr0 + (lane >> 2)) * NN + (b & 7) * 512 + (lane & 3) * 128
                          : n0 + rank * KC * NN;
        const int pfs = pr0 < KC ? KC * NN : 0;
#define PFL "v_and_b32 %[pf], 0xffffff, %[pf]\n\tglobal_load_dword v251, %[pf], %[w]\n\tv_add_u32 %[pf], %[pfs], %[pf]\n\t"
#define XST(q, r, off) "global_store_dwordx4 %[" #q "], v[" #r ":" #r "+3], %[peer] offset:" #off " sc0 sc1\n\t"
#define XST8(q, r) XST(q, r, -4096) XST(q, r+4, -3072) XST(q, r+8, -2048) XST(q, r+12, -1024) \
                   XST(q, r+16, 0) XST(q, r+20, 1024) XST(q, r+24, 2048) XST(q, r+28, 3072)
#define FLG(f) "global_store_dword %[" #f "], %[call], %[peer] sc0 sc1\n\t"
        int qo = qoff(slot);
        unsigned tgo = slot * PACE2 / 2 - 10;
        unsigned callv;
        if (serve) {
#pragma unroll 1
            for (int i = 0; i < 2; ++i) {
                asm volatile(PFL PFL PFL PFL PFL PFL PFL : [pf] "+v"(pf) : [w] "s"(pwm), [pfs] "s"(pfs) : "memory", "v251");
                pf += (KC + 128) * NN;
            }
            asm volatile("s_barrier\n\ts_waitcnt vmcnt(7)\n\tv_add_u32 %0, 1, v252" : "=v"(callv) :: "memory");
        } else if (multi) {
            XLOAD(160, slot + 4);
            XLOAD(192, slot + 8);
            int q1 = qoff(slot + 4), q2 = qoff(slot + 8), tmp;
#define SPIN(L) L "%=:\n\ts_memrealtime s[60:61]\n\ts_waitcnt lgkmcnt(0)\n\ts_sub_i32 %[tmp], s60, %[tgo]\n\t" \
                "s_cmp_lt_i32 %[tmp], 0\n\ts_cbranch_scc1 " L "%=\n\ts_add_u32 %[tgo], %[tgo], %[gap]\n\t"
            asm volatile(
                "s_waitcnt vmcnt(17)\n\t"
                "s_memrealtime s[60:61]\n\t"
                "v_add_u32 %[call], 1, v252\n\t"
                "v_and_b32 v252, 1, %[call]\n\t"
                "v_lshl_add_u32 %[q0], v252, 21, %[q0]\n\t"
                "v_lshl_add_u32 %[q1], v252, 21, %[q1]\n\t"
                "v_lshl_add_u32 %[q2], v252, 21, %[q2]\n\t"
                "s_waitcnt lgkmcnt(0)\n\t"
                "s_add_u32 %[tgo], %[tgo], s60\n\t"
                SPIN("Lm0") XST8(q0, 128)
                "s_waitcnt vmcnt(8)\n\t"
                SPIN("Lm1") XST8(q1, 160)
                "s_waitcnt vmcnt(8)\n\t" FLG(f0)
                SPIN("Lm2") XST8(q2, 192)
                "s_waitcnt vmcnt(9)\n\t" FLG(f1)
                "s_waitcnt vmcnt(1)\n\t" FLG(f2)
                "s_barrier"
                : [call] "=&v"(callv), [q0] "+v"(qo), [q1] "+v"(q1), [q2] "+v"(q2), [tgo] "+s"(tgo), [tmp] "=&s"(tmp)
                : [f0] "v"(fpeer(slot)), [f1] "v"(fpeer(slot + 4)), [f2] "v"(fpeer(slot + 8)), [peer] "s"(peer),
                  [gap] "n"(2 * PACE2)
                : "memory", "scc", "v252", "s60", "s61");
        } else {
            asm volatile(
                "s_waitcnt vmcnt(1)\n\t"
                "s_memrealtime s[60:61]\n\t"
                "v_add_u32 %[call], 1, v252\n\t"
                "v_and_b32 v252, 1, %[call]\n\t"
                "v_lshl_add_u32 %[qo], v252, 21, %[qo]\n\t"
                "s_waitcnt lgkmcnt(0)\n\t"
                "s_add_u32 %[tgo], %[tgo], s60"
                : [call] "=&v"(callv), [qo] "+v"(qo), [tgo] "+s"(tgo) :: "memory", "scc", "v252", "s60", "s61");
        }
        int ready, tau, tmp;
#define G_POLL(L) \
        "s_getreg_b32 %[tmp], hwreg(HW_REG_IB_STS)\n\t" \
        "s_and_b32 %[tmp], %[tmp], 0xc0000f\n\t" \
        "s_cbranch_scc1 " L "b%=\n\t" \
        "s_waitcnt vmcnt(0)\n\t" \
        "v_cmp_le_u32 vcc, %[call], v250\n\t" \
        "s_or_b64 vcc, vcc, %[known]\n\t" \
        "s_ff0_i32_b64 %[ready], vcc\n\t" \
        "s_bitcmp1_b32 vcc_hi, 31\n\t" \
        "s_cbranch_scc1 Ldead%=\n\t" \
        "s_sub_i32 %[tmp], %[ready], %[tau]\n\t" \
        "s_cmp_lt_i32 %[tmp], %[hor]\n\t" \
        "s_cbranch_scc0 " L "b%=\n\t" \
        "global_load_dword v250, %[foff], %[my] sc0 sc1\n\t" \
        L "b%=:\n\t"
#define G_TICK(L, NOBAR) \
        "s_cmp_le_i32 %[tau], %[ready]\n\t" \
        "s_cbranch_scc0 " NOBAR "%=\n\t" \
        "s_barrier\n\t" \
        "s_add_i32 %[tau], %[tau], 1\n\t" \
        "s_branch " L "%=\n\t"
        asm volatile(
            "s_mov_b32 %[ready], -1\n\t"
            "s_mov_b32 %[tau], 0\n\t"
            "s_cmp_lg_u32 %[done], 0\n\t"
            "s_cbranch_scc1 Lc%=\n\t"
            "La%=:\n\t"
            "s_memrealtime s[60:61]\n\t"
            G_POLL("La")
            "s_min_i32 %[ready], %[ready], %[last]\n\t"
            "s_waitcnt lgkmcnt(0)\n\t"
            "s_sub_i32 %[tmp], s60, %[tgo]\n\t"
            "s_cmp_ge_i32 %[tmp], 0\n\t"
            "s_cbranch_scc1 Lpush%=\n\t"
            G_TICK("La", "La")
            "Lpush%=:\n\t"
            "s_cmp_eq_u32 %[push], 0\n\t"
            "s_cbranch_scc1 Lnost%=\n\t"
            XST8(qo, 128)
            "Lnost%=:\n\t"
            PFL PFL PFL
            "s_waitcnt vmcnt(3)\n\t"
            "v_cmp_le_u32 vcc, %[call], v250\n\t"
            "s_cbranch_scc1 Lnofl%=\n\t"
            FLG(f0)
            "Lnofl%=:\n\t"
            PFL PFL PFL PFL
            "global_load_dword v250, %[foff], %[my] sc0 sc1\n\t"
            "s_or_b64 vcc, vcc, %[known]\n\t"
            "s_bitset0_b32 vcc_hi, 31\n\t"
            "s_ff0_i32_b64 %[ready], vcc\n\t"
            "Lc%=:\n\t"
            G_POLL("Lc")
            "s_cmp_gt_i32 %[ready], %[last]\n\t"
            "s_cbranch_scc1 Ld%=\n\t"
            G_TICK("Lc", "Lcw")
            "Lcw%=:\n\t"
            "s_memrealtime s[60:61]\n\t"
            "s_waitcnt lgkmcnt(0)\n\t"
            "s_sub_i32 %[tmp], s60, %[tdead]\n\t"
            "s_cmp_lt_i32 %[tmp], 0\n\t"
            "s_cbranch_scc1 Lc%=\n\t"
            "Ldead%=:\n\t"
            "global_store_dword %[tmo], %[call], %[my] sc0 sc1\n\t"
            "ds_write_b32 %[ctl], %[call]\n\t"
            "s_waitcnt lgkmcnt(0)\n\t"
            "Ld%=:"
            : [ready] "=&s"(ready), [tau] "=&s"(tau), [tmp] "=&s"(tmp), [pf] "+v"(pf)
            : [call] "v"(callv), [foff] "v"(foff), [qo] "v"(qo), [f0] "v"(fpeer(slot)), [tmo] "v"(tmo),
              [ctl] "v"((int)(unsigned)(size_t)L + L_CTRL),
              [w] "s"(pwm), [my] "s"(my), [peer] "s"(peer), [known] "s"(KNOWN), [push] "s"(push), [pfs] "s"(pfs),
              [done] "s"(__builtin_amdgcn_readfirstlane(multi || serve)),
              [tgo] "s"(__builtin_amdgcn_readfirstlane(tgo)), [tdead] "s"((unsigned)tstart + DEADLINE_TICKS),
              [last] "n"(NBAR - 1), [hor] "n"(HORIZON)
            : "memory", "vcc", "scc", "v250", "v251", "s60", "s61");
        return;
    }

    if (wave < 4) {
        __builtin_amdgcn_s_setprio(2);
        if (wave == 1) {
            const __amdgpu_buffer_rsrc_t rx =
                __builtin_amdgcn_make_buffer_rsrc((void*)px, (short)0, NR * MS * KC, 0x00020000);
            int voff[4];
#pragma unroll
            for (int u = 0; u < 4; ++u) voff[u] = ((u & 1) ? (xoff ^ 64) : xoff) + u * (8 * KC - 1024);
            asm volatile("s_barrier\n\tglobal_load_dword v252, %0, %1 sc0 sc1"
                         :: "v"(0), "s"(my + WS_CNT + (size_t)b * 128) : "memory", "v252");
#pragma unroll
            for (int i = 0; i < 4; ++i) {
                const int soff = (rank * MS + mt * 64) * KC + ((i + mt + nx) & 3) * 128;
                DMA4(rx, L + L_A + i * A_SLOT, voff[0], voff[1], voff[2], voff[3], soff, 0);
                DMA4(rx, L + L_A + i * A_SLOT + 4096, voff[0], voff[1], voff[2], voff[3], soff + 32 * KC, 0);
                if (i >= 1) { WVX(8); BAR(); }
            }
            unsigned cv;
            asm volatile("s_waitcnt vmcnt(0)\n\tv_add_u32 %0, 1, v252\n\tds_write_b32 %1, %0 offset:4"
                         : "=&v"(cv) : "v"((int)(unsigned)(size_t)L + L_CTRL) : "memory");
            const unsigned call = __builtin_amdgcn_readfirstlane(cv);
            const __amdgpu_buffer_rsrc_t ra = __builtin_amdgcn_make_buffer_rsrc(
                (void*)(my + WS_A + (size_t)(call & 1) * A_BYTES + (size_t)mt * 32 * A_SLOT), (short)0,
                32 * A_SLOT, 0x00020000);
#pragma unroll
            for (int t = 3; t <= T0 - 1 - XD; ++t) BAR();
            asm volatile("s_waitcnt lgkmcnt(0)" ::: "memory");
            static_assert(XD == 3, "waits below");
            int lo = L_A + (4 % NA) * A_SLOT;
#pragma unroll 1
            for (int s = 4; s < NSTEP; ++s) {
                const int soff = __builtin_amdgcn_readlane(tab, s) * A_SLOT;
                DMA4(ra, L + lo, lane * 16, lane * 16, lane * 16, lane * 16, soff, 17);
                DMA4(ra, L + lo + 4096, lane * 16, lane * 16, lane * 16, lane * 16, soff + 4096, 17);
                lo = lo == L_A + (NA - 1) * A_SLOT ? L_A : lo + A_SLOT;
                WVX(16); BAR();
            }
            WVX(8); BAR();
            WVX(0); BAR();
            return;
        }
        const int ld = wave - 2;
        const __amdgpu_buffer_rsrc_t rw =
            __builtin_amdgcn_make_buffer_rsrc((void*)pwm, (short)0, KK * NN, 0x00020000);
        int voff[4];
#pragma unroll
        for (int u = 0; u < 4; ++u) voff[u] = (u * 8 + (lane >> 3)) * NN - u * 1024 + (swz ^ ((u >> 1) << 6));
        const int so = (tab * 128 + ld * 64) * NN + n0;
#define W_HALF(l, h, sl) DMA4(rw, L + (sl) * W_SLOT + ld * 8192 + (h) * 4096, voff[0], voff[1], voff[2], voff[3], \
                              __builtin_amdgcn_readlane(so, l) + (h) * 32 * NN, 0)
        BAR();
        W_SCHEDULE
        return;
    }

    const int cw = wave - 4;
    if (cw == 0) *(volatile LAS unsigned*)(L + L_CTRL) = 0u;
    const int g4 = lane >> 4, l15 = lane & 15, q8 = l15 >> 1;
    const int lbase = (int)(unsigned)(size_t)L;
    const int trb = lbase + (g4 * 16 + q8) * 128 + (((cw * 2) ^ (q8 >> 1) ^ ((g4 & 1) << 2)) * 16) + (lane & 1) * 8;
    const int arb = lbase + L_A + l15 * 128 + ((g4 ^ (l15 >> 1)) * 16);
    float* ob = pout + (size_t)(mt * 64) * NN + n0 + cw * 32;
    float sx, sw;
    asm volatile(
        COMPUTE_ASM
        : [sx] "=&s"(sx), [sw] "=&s"(sw)
        : [wb0] "v"(trb), [wb1] "v"(trb ^ 16), [wb0h] "v"(trb + 65536), [wb1h] "v"((trb ^ 16) + 65536),
          [ab0] "v"(arb), [ab1] "v"(arb ^ 64), [oo] "v"(l15 * NN * 4 + g4 * 16), [ctl] "s"(lbase + L_CTRL),
          [psx] "s"(psx), [psw] "s"(psw), [cntp] "s"(my + WS_CNT + (size_t)b * 128), [ob0] "s"(ob),
          [ob1] "s"(ob + 16 * NN), [ob2] "s"(ob + 32 * NN), [ob3] "s"(ob + 48 * NN)
        : "memory", "vcc", CLOBBER_16_255);
}

extern "C" void kernel_launch(int rank, int nranks, ncclComm_t comm, hipStream_t stream,
                              void* const* d_in, const int* in_sizes, int n_in,
                              void* d_out, int out_size, const int* in_dtypes, int out_dtype,
                              void* const* d_out_all, void* const* d_ws_all, size_t ws_bytes) {
    (void)comm; (void)in_dtypes; (void)out_dtype; (void)d_out_all;
    if (nranks != NR || (unsigned)rank >= (unsigned)NR || n_in < 4 || ws_bytes < WS_NEED ||
        out_size != MS * NN || in_sizes[0] != NR * MS * KC || in_sizes[1] != KK * NN) {
        fprintf(stderr, "a2a_gemm: unexpected problem shape\n");
        return;
    }
    HIP_CHECK(hipFuncSetAttribute((const void*)a2a_gemm_kernel,
                                  hipFuncAttributeMaxDynamicSharedMemorySize, LDS_BYTES));
    Params p;
    for (int i = 0; i < NR; ++i) p.ws[i] = (unsigned char*)d_ws_all[i];
    p.x = (const unsigned char*)d_in[0];
    p.w = (const unsigned char*)d_in[1];
    p.my = (unsigned char*)d_ws_all[rank];
    p.out = (float*)d_out;
    p.sx = (const float*)d_in[2];
    p.sw = (const float*)d_in[3];
    p.rank = rank;
    p.pad = 0;
    hipLaunchKernelGGL(a2a_gemm_kernel, dim3(NWG), dim3(NTHREADS), LDS_BYTES, stream, p);
    HIP_CHECK(hipGetLastError());
}
```
